```python
import numpy as np

kernel_name = "gpu_gemm_rs_m4096_k4096_n16384_f32_relu"


ABSMAX_RTOL = 0.001

COMPUTE_DTYPE = "float32"
EPILOGUE = "relu"
IN_QUANT = None
OUT_QUANT = None
IN_DTYPES = {"x": "float32", "w_mat": "float32"}
WORLD_SIZE = 8

def setup_inputs():

    x = np.random.randn(4096, 4096).astype(np.float32)
    w_mat = (np.random.randn(4096, 16384) / np.sqrt(4096)).astype(np.float32)
    return {"x": x, "w_mat": w_mat}

def _epilogue(y):
    return np.maximum(y, 0.0).astype(np.float32)

def reference(x, w_mat):
    y = (x @ w_mat).astype(np.float32)
    return _epilogue(y)
```

```cpp
#include <hip/hip_runtime.h>
#include <rccl/rccl.h>
#include <cstdio>

#define HIP_CHECK(expr) do { hipError_t _e = (expr); if (_e != hipSuccess) { \
    fprintf(stderr, "HIP error %s at %s:%d\n", hipGetErrorString(_e), __FILE__, __LINE__); \
    return; } } while (0)

namespace {

#define LAS __attribute__((address_space(3)))
typedef _Float16 h8 __attribute__((ext_vector_type(8)));
typedef float f32x4 __attribute__((ext_vector_type(4)));

constexpr int NR = 8;
constexpr int MTOT = 4096;
constexpr int KS = 512;
constexpr int NTOT = 16384;
constexpr int NQ = NTOT / NR;
constexpr int MSH = MTOT / NR;
constexpr int BM = 256, BK = 64, HALF = 128;
constexpr int HTB = HALF * BK * 2;
constexpr int STAGE_BYTES = 8 * HTB;
constexpr int NMT = MTOT / BM;
constexpr int NNT = NQ / BM;
constexpr int NCH = KS / BK;
constexpr int NKT = NR * NCH;
constexpr int LDS_BYTES = STAGE_BYTES + 256;

constexpr size_t OFF_CTR = 0;
constexpr size_t OFF_TMO = 256;
constexpr size_t OFF_FX = 4096;
constexpr size_t OFF_FW = OFF_FX + (size_t)NR * NCH * NMT * 128;
constexpr size_t OFF_DONE = OFF_FW + (size_t)NR * NCH * NNT * 128;
constexpr size_t OFF_STAGE = 1ull << 20;
constexpr size_t XA_BYTES = (size_t)NMT * NKT * 2 * HTB;
constexpr size_t WB_BYTES = (size_t)NNT * NKT * 2 * HTB;
constexpr size_t PAR_BYTES = XA_BYTES + WB_BYTES;
constexpr size_t WS_NEED = OFF_STAGE + 2 * PAR_BYTES;
static_assert(OFF_DONE + (size_t)NR * 16 * 128 <= OFF_STAGE, "flag region");

constexpr unsigned long long DEADLINE_TICKS = 40000000ull;

struct Params {
    const float* x;
    const float* w;
    float* out[NR];
    unsigned char* ws[NR];
    int rank;
};

__device__ __forceinline__ int lds_byte(int r, int c) {
    const int st = (r >> 4) * 2 + (c >> 5), rr = r & 15, cc = c & 31, ob = rr * 64 + cc * 2;
    return st * 1024 + (ob ^ (((ob >> 9) & 1) << 5));
}
__device__ __forceinline__ void stage_rc(int b, int& R, int& C) {
    const int st = b / 1024, sb = b % 1024, swz = sb ^ (((sb >> 9) & 1) << 5);
    R = (st >> 1) * 16 + swz / 64; C = (st & 1) * 32 + (swz % 64) / 2;
}

__device__ __forceinline__ unsigned ld_sys(const unsigned* p) {
    return __hip_atomic_load(p, __ATOMIC_RELAXED, __HIP_MEMORY_SCOPE_SYSTEM);
}

__device__ __forceinline__ h8 cvt8(const f32x4 a, const f32x4 b) {
    h8 r;
    r[0] = (_Float16)a[0]; r[1] = (_Float16)a[1]; r[2] = (_Float16)a[2]; r[3] = (_Float16)a[3];
    r[4] = (_Float16)b[0]; r[5] = (_Float16)b[1]; r[6] = (_Float16)b[2]; r[7] = (_Float16)b[3];
    return r;
}

__device__ __forceinline__ void publish(unsigned* flag, unsigned call) {
    asm volatile("s_waitcnt vmcnt(0)" ::: "memory");
    __syncthreads();
    if (threadIdx.x == 0) {
        __builtin_amdgcn_fence(__ATOMIC_RELEASE, "");
        asm volatile("s_waitcnt vmcnt(0)" ::: "memory");
        __hip_atomic_store(flag, call, __ATOMIC_RELAXED, __HIP_MEMORY_SCOPE_SYSTEM);
    }
}

__global__ void __launch_bounds__(512, 2) fused_kernel(Params p) {
    extern __shared__ __attribute__((aligned(16))) unsigned char smem[];
    LAS unsigned char* lds = (LAS unsigned char*)smem;
    const int tid = threadIdx.x;
    const int rank = p.rank;
    unsigned char* myws = p.ws[rank];
    unsigned* tmo = (unsigned*)(myws + OFF_TMO);

    if (tid == 0) {
        unsigned tk = __hip_atomic_fetch_add((unsigned*)(myws + OFF_CTR), 1u, __ATOMIC_RELAXED, __HIP_MEMORY_SCOPE_AGENT);
        *(LAS unsigned*)(lds + STAGE_BYTES) = tk / gridDim.x + 1u;
    }
    __syncthreads();
    const unsigned call = __builtin_amdgcn_readfirstlane(*(LAS unsigned*)(lds + STAGE_BYTES));
    const size_t par_off = OFF_STAGE + (size_t)(call & 1u) * PAR_BYTES;

    const int xcd = blockIdx.x & 7, idx = blockIdx.x >> 3;
    if (idx < 16) {
        const int pid = idx * 8 + xcd;
        const int wid = __builtin_amdgcn_readfirstlane(tid >> 6), lane = tid & 63;
        for (int u = pid; u < NCH * 192; u += 128) {
            const int j = u / 192, v = u % 192;
            const int t = j * NR + rank;
            const int dofs = v & 7;
            const int d = (rank + 1 + dofs) & 7;
            const int tile = v >> 3;
            unsigned char* dws = p.ws[d];
            if (tile < NMT) {
                const int mt = tile;
                unsigned char* dst = dws + par_off + ((size_t)(mt * NKT + t) * 2) * HTB;
                f32x4 a[4][2];
#pragma unroll
                for (int i = 0; i < 4; ++i) {
                    const int b = (i * 512 + tid) * 16;
                    int R, C; stage_rc(b & (HTB - 1), R, C);
                    const float* src = p.x + (size_t)(mt * BM + (b >> 14) * HALF + R) * KS + j * BK + C;
                    a[i][0] = *(const f32x4*)src; a[i][1] = *(const f32x4*)(src + 4);
                }
#pragma unroll
                for (int i = 0; i < 4; ++i) {
                    const int b = (i * 512 + tid) * 16;
                    *(h8*)(dst + b) = cvt8(a[i][0], a[i][1]);
                }
                publish((unsigned*)(dws + OFF_FX + ((size_t)((rank * NCH + j) * NMT + mt)) * 128), call);
            } else {
                const int nt = tile - NMT;
                unsigned char* dst = dws + par_off + XA_BYTES + ((size_t)(nt * NKT + t) * 2) * HTB;
                const int k0 = (wid & 1) * 32 + (lane >> 4) * 8;
                const int n0 = (wid >> 1) * 64 + (lane & 15) * 4;
                const float* src = p.w + (size_t)(j * BK + k0) * NTOT + d * NQ + nt * BM + n0;
                f32x4 a[8];
#pragma unroll
                for (int jj = 0; jj < 8; ++jj) a[jj] = *(const f32x4*)(src + (size_t)jj * NTOT);
#pragma unroll
                for (int i = 0; i < 4; ++i) {
                    h8 r;
#pragma unroll
                    for (int jj = 0; jj < 8; ++jj) r[jj] = (_Float16)a[jj][i];
                    const int n = n0 + i;
                    *(h8*)(dst + (n >> 7) * HTB + lds_byte(n & 127, k0)) = r;
                }
                publish((unsigned*)(dws + OFF_FW + ((size_t)((rank * NCH + j) * NNT + nt)) * 128), call);
            }
        }
        if (blockIdx.x == 0 && tid < 64) {
            const unsigned* done = (const unsigned*)(myws + OFF_DONE);
            const unsigned long long t0 = __builtin_amdgcn_s_memrealtime();
            bool bad = false;
            for (;;) {
                const int e0 = lane, e1 = lane + 64;
                unsigned v0 = ((e0 >> 4) == rank) ? call : ld_sys(done + e0 * 32);
                unsigned v1 = ((e1 >> 4) == rank) ? call : ld_sys(done + e1 * 32);
                if (__all(v0 >= call && v1 >= call)) break;
                if (__hip_atomic_load(tmo, __ATOMIC_RELAXED, __HIP_MEMORY_SCOPE_AGENT) == call ||
                    __builtin_amdgcn_s_memrealtime() - t0 > DEADLINE_TICKS) {
                    __hip_atomic_store(tmo, call, __ATOMIC_RELAXED, __HIP_MEMORY_SCOPE_AGENT);
                    bad = true; break;
                }
                __builtin_amdgcn_s_sleep(2);
            }
            __builtin_amdgcn_fence(__ATOMIC_ACQUIRE, "");
            if (bad && lane == 0) p.out[rank][0] = __builtin_nanf("");
        }
        return;
    }

    const int g = idx - 16;
    const int mt = (xcd >> 1) * 4 + (g >> 2);
    const int nt = (xcd & 1) * 4 + (g & 3);
    const int wid = __builtin_amdgcn_readfirstlane(tid >> 6), lane = tid & 63, wr = wid >> 2, wc = wid & 3, fr = lane & 15, fq = lane >> 4;
    const unsigned voff = (unsigned)tid * 16u;
    const unsigned ldsw = (unsigned)wid * 1024u;
    const int aoff = lds_byte(wr * 64 + fr, fq * 8), boff = lds_byte(wc * 32 + fr, fq * 8);
    const unsigned char* cA = myws + par_off + (size_t)mt * NKT * 2 * HTB;
    const unsigned char* cB = myws + par_off + XA_BYTES + (size_t)nt * NKT * 2 * HTB;
    const unsigned* fxp = (const unsigned*)(myws + OFF_FX) + (size_t)mt * 32;
    const unsigned* fwp = (const unsigned*)(myws + OFF_FW) + (size_t)nt * 32;

#define SA(b, h) (((b) * 2 + (h)) * HTB)
#define SB(b, h) ((4 + (b) * 2 + (h)) * HTB)
#define STAGE(bufoff, gbase) do { _Pragma("unroll") for (int _i = 0; _i < 2; ++_i) \
        __builtin_amdgcn_global_load_lds((const unsigned*)((gbase) + voff + _i * 8192), (LAS unsigned*)(lds + (bufoff) + ldsw + _i * 8192), 16, 0, 0); } while (0)
#define LDA(dst, b, h) do { _Pragma("unroll") for (int m = 0; m < 4; ++m) _Pragma("unroll") for (int k = 0; k < 2; ++k) dst[m][k] = *(const LAS h8*)(lds + SA(b, h) + aoff + m * 2048 + k * 1024); } while (0)
#define LDB(dst, b, h) do { _Pragma("unroll") for (int n = 0; n < 2; ++n) _Pragma("unroll") for (int k = 0; k < 2; ++k) dst[n][k] = *(const LAS h8*)(lds + SB(b, h) + boff + n * 2048 + k * 1024); } while (0)
#define MMA(ai, bj, At, Bt) do { __builtin_amdgcn_s_setprio(1); _Pragma("unroll") for (int m = 0; m < 4; ++m) _Pragma("unroll") for (int n = 0; n < 2; ++n) _Pragma("unroll") for (int k = 0; k < 2; ++k) \
        acc[ai][bj][m][n] = __builtin_amdgcn_mfma_f32_16x16x32_f16(Bt[n][k], At[m][k], acc[ai][bj][m][n], 0, 0, 0); __builtin_amdgcn_s_setprio(0); } while (0)
#define WAIT_V(n) asm volatile("s_waitcnt vmcnt(" #n ")" ::: "memory")
#define WAIT_L(n) asm volatile("s_waitcnt lgkmcnt(" #n ")" ::: "memory")
#define BAR __builtin_amdgcn_s_barrier()
#define SCHED __builtin_amdgcn_sched_barrier(0)
#define READY(j) do { \
        if (tid < 64) { \
            const unsigned* fp = (lane < 8) ? fxp + (size_t)((lane * NCH + (j)) * NMT) * 32 : fwp + (size_t)(((lane & 7) * NCH + (j)) * NNT) * 32; \
            const unsigned long long t0 = __builtin_amdgcn_s_memrealtime(); \
            for (;;) { \
                unsigned v = (lane < 16) ? ld_sys(fp) : call; \
                if (__all(v >= call)) break; \
                if (__hip_atomic_load(tmo, __ATOMIC_RELAXED, __HIP_MEMORY_SCOPE_AGENT) == call || \
                    __builtin_amdgcn_s_memrealtime() - t0 > DEADLINE_TICKS) { \
                    __hip_atomic_store(tmo, call, __ATOMIC_RELAXED, __HIP_MEMORY_SCOPE_AGENT); break; } \
                __builtin_amdgcn_s_sleep(2); \
            } \
            __builtin_amdgcn_fence(__ATOMIC_ACQUIRE, ""); \
            asm volatile("s_waitcnt vmcnt(0)" ::: "memory"); \
        } \
        asm volatile("" ::: "memory"); __builtin_amdgcn_s_barrier(); asm volatile("" ::: "memory"); \
    } while (0)

    f32x4 acc[2][2][4][2];
#pragma unroll
    for (int a = 0; a < 2; ++a)
#pragma unroll
        for (int b = 0; b < 2; ++b)
#pragma unroll
            for (int m = 0; m < 4; ++m)
#pragma unroll
                for (int n = 0; n < 2; ++n) acc[a][b][m][n] = (f32x4){0.f, 0.f, 0.f, 0.f};
    h8 At[4][2], B0[2][2], B1[2][2];
    constexpr size_t kstep = 2 * HTB, hstep = HTB;

    READY(0);
    STAGE(SB(0, 0), cB); STAGE(SA(0, 0), cA); STAGE(SB(0, 1), cB + hstep); STAGE(SA(0, 1), cA + hstep);
    if (wr == 1) BAR;
    WAIT_V(4); BAR;
    STAGE(SB(1, 0), cB + kstep); STAGE(SA(1, 0), cA + kstep); STAGE(SB(1, 1), cB + hstep + kstep);
    WAIT_V(6); BAR;
    for (int t = 0; t < NKT; t += 2) {
        const bool last = (t == NKT - 2);
        const unsigned char* a1 = cA + (size_t)(t + 1) * kstep;
        const unsigned char* a2 = last ? cA : cA + (size_t)(t + 2) * kstep;
        const unsigned char* b2 = last ? cB : cB + (size_t)(t + 2) * kstep;
        const unsigned char* a3 = a2 + kstep; const unsigned char* b3 = b2 + kstep;
        if (!last && ((t + 2) & 7) == 0) READY((t + 2) >> 3);
        LDB(B0, 0, 0); SCHED; LDA(At, 0, 0); STAGE(SA(1, 1), a1 + hstep);
        WAIT_L(8); BAR; WAIT_L(0); MMA(0, 0, At, B0); BAR; SCHED;
        LDB(B1, 0, 1); STAGE(SB(0, 0), b2);
        BAR; WAIT_L(0); MMA(0, 1, At, B1); BAR;
        LDA(At, 0, 1); STAGE(SA(0, 0), a2);
        BAR; WAIT_L(0); MMA(1, 0, At, B0); BAR; SCHED;
        STAGE(SB(0, 1), b2 + hstep);
        WAIT_V(6); BAR; MMA(1, 1, At, B1); BAR;
        LDB(B0, 1, 0); SCHED; LDA(At, 1, 0); STAGE(SA(0, 1), a2 + hstep);
        WAIT_L(8); BAR; WAIT_L(0); MMA(0, 0, At, B0); BAR; SCHED;
        LDB(B1, 1, 1); STAGE(SB(1, 0), b3);
        BAR; WAIT_L(0); MMA(0, 1, At, B1); BAR;
        LDA(At, 1, 1); STAGE(SA(1, 0), a3);
        BAR; WAIT_L(0); MMA(1, 0, At, B0); BAR; SCHED;
        STAGE(SB(1, 1), b3 + hstep);
        WAIT_V(6); BAR; MMA(1, 1, At, B1); BAR;
    }
    WAIT_V(0);
    if (wr == 0) BAR;
    BAR;

    const bool bad = (__hip_atomic_load(tmo, __ATOMIC_RELAXED, __HIP_MEMORY_SCOPE_AGENT) == call);
    const int dest = mt >> 1;
    float* obase = p.out[dest] + (size_t)((mt & 1) * BM + wr * 64 + fr) * NTOT + rank * NQ + nt * BM + wc * 32 + 4 * fq;
    const float qn = __builtin_nanf("");
#pragma unroll
    for (int ai = 0; ai < 2; ++ai)
#pragma unroll
        for (int m = 0; m < 4; ++m) {
            float* rowp = obase + (size_t)(ai * HALF + m * 16) * NTOT;
#pragma unroll
            for (int bj = 0; bj < 2; ++bj)
#pragma unroll
                for (int n = 0; n < 2; ++n) {
                    f32x4 v = acc[ai][bj][m][n];
                    v[0] = v[0] > 0.f ? v[0] : 0.f; v[1] = v[1] > 0.f ? v[1] : 0.f;
                    v[2] = v[2] > 0.f ? v[2] : 0.f; v[3] = v[3] > 0.f ? v[3] : 0.f;
                    if (bad) v = (f32x4){qn, qn, qn, qn};
                    *(f32x4*)(rowp + bj * HALF + n * 16) = v;
                }
        }
    if (dest != rank)
        publish((unsigned*)(p.ws[dest] + OFF_DONE + (size_t)(rank * 16 + (mt & 1) * 8 + nt) * 128), call);
}

}

extern "C" void kernel_launch(int rank, int nranks, ncclComm_t comm,
                              hipStream_t stream,
                              void* const* d_in, const int* in_sizes, int n_in,
                              void* d_out, int out_size,
                              const int* in_dtypes, int out_dtype,
                              void* const* d_out_all,
                              void* const* d_ws_all, size_t ws_bytes) {
    if (nranks != NR || ws_bytes < WS_NEED) { fprintf(stderr, "unsupported configuration\n"); return; }
    static bool init = false;
    if (!init) {
        HIP_CHECK(hipFuncSetAttribute((const void*)fused_kernel, hipFuncAttributeMaxDynamicSharedMemorySize, LDS_BYTES));
        init = true;
    }
    Params p;
    p.x = (const float*)d_in[0];
    p.w = (const float*)d_in[1];
    for (int i = 0; i < NR; ++i) { p.out[i] = (float*)d_out_all[i]; p.ws[i] = (unsigned char*)d_ws_all[i]; }
    p.rank = rank;
    fused_kernel<<<256, 512, LDS_BYTES, stream>>>(p);
    HIP_CHECK(hipGetLastError());
}
```

```cpp
#include <hip/hip_runtime.h>
#include <rccl/rccl.h>
#include <cstdio>

#define HIP_CHECK(expr) do { hipError_t _e = (expr); if (_e != hipSuccess) { \
    fprintf(stderr, "HIP error %s at %s:%d\n", hipGetErrorString(_e), __FILE__, __LINE__); \
    return; } } while (0)

namespace {

#define LAS __attribute__((address_space(3)))
typedef _Float16 h8 __attribute__((ext_vector_type(8)));
typedef float f32x4 __attribute__((ext_vector_type(4)));

constexpr int NR = 8;
constexpr int MTOT = 4096;
constexpr int KS = 512;
constexpr int NTOT = 16384;
constexpr int NQ = NTOT / NR;
constexpr int MSH = MTOT / NR;
constexpr int BM = 256, BK = 64, HALF = 128;
constexpr int HTB = HALF * BK * 2;
constexpr int STAGE_BYTES = 8 * HTB;
constexpr int NMT = MTOT / BM;
constexpr int NNT = NQ / BM;
constexpr int NCH = KS / BK;
constexpr int NKT = NR * NCH;
constexpr int LDS_BYTES = STAGE_BYTES + 256;

constexpr size_t OFF_CTR = 0;
constexpr size_t OFF_TMO = 256;
constexpr size_t OFF_QCTR = 512;
constexpr int NUNITS = NCH * 192;
#define NDEF 5
#define NPX 16
constexpr int NPUSH = NPX * 8;
constexpr size_t OFF_FX = 4096;
constexpr size_t OFF_FW = OFF_FX + (size_t)NR * NCH * NMT * 128;
constexpr size_t OFF_DONE = OFF_FW + (size_t)NR * NCH * NNT * 128;
constexpr size_t OFF_STAGE = 1ull << 20;
constexpr size_t XA_BYTES = (size_t)NMT * NKT * 2 * HTB;
constexpr size_t WB_BYTES = (size_t)NNT * NKT * 2 * HTB;
constexpr size_t PAR_BYTES = XA_BYTES + WB_BYTES;
constexpr size_t WS_NEED = OFF_STAGE + 2 * PAR_BYTES;
static_assert(OFF_DONE + (size_t)NR * 16 * 128 <= OFF_STAGE, "flag region");

#define STAMP() do {} while (0)
#define STAMP_INIT()
constexpr unsigned long long DEADLINE_TICKS = 40000000ull;

struct Params {
    const float* x;
    const float* w;
    float* out[NR];
    unsigned char* ws[NR];
    int rank;
};

__device__ __forceinline__ int lds_byte(int r, int c) {
    const int st = (r >> 4) * 2 + (c >> 5), rr = r & 15, cc = c & 31, ob = rr * 64 + cc * 2;
    return st * 1024 + (ob ^ (((ob >> 9) & 1) << 5));
}
__device__ __forceinline__ void stage_rc(int b, int& R, int& C) {
    const int st = b / 1024, sb = b % 1024, swz = sb ^ (((sb >> 9) & 1) << 5);
    R = (st >> 1) * 16 + swz / 64; C = (st & 1) * 32 + (swz % 64) / 2;
}

__device__ __forceinline__ unsigned ld_sys(const unsigned* p) {
    return __hip_atomic_load(p, __ATOMIC_RELAXED, __HIP_MEMORY_SCOPE_SYSTEM);
}

__device__ __forceinline__ h8 cvt8(const f32x4 a, const f32x4 b) {
    h8 r;
    r[0] = (_Float16)a[0]; r[1] = (_Float16)a[1]; r[2] = (_Float16)a[2]; r[3] = (_Float16)a[3];
    r[4] = (_Float16)b[0]; r[5] = (_Float16)b[1]; r[6] = (_Float16)b[2]; r[7] = (_Float16)b[3];
    return r;
}

template <class T> __device__ __forceinline__ void st_wt(void* ptr, const T v) {
    asm volatile("global_store_dwordx4 %0, %1, off sc0 sc1\n\ts_nop 1" :: "v"(ptr), "v"(v) : "memory");
}

__device__ __forceinline__ void publish(unsigned* flag, unsigned call) {
    asm volatile("s_waitcnt vmcnt(0)" ::: "memory");
    __syncthreads();
    if (threadIdx.x == 0) {
        __hip_atomic_store(flag, call, __ATOMIC_RELAXED, __HIP_MEMORY_SCOPE_SYSTEM);
    }
}

__global__ void __launch_bounds__(512, 2) fused_kernel(Params p) {
    extern __shared__ __attribute__((aligned(16))) unsigned char smem[];
    LAS unsigned char* lds = (LAS unsigned char*)smem;
    const int tid = threadIdx.x;
    const int rank = p.rank;
    unsigned char* myws = p.ws[rank];
    unsigned* tmo = (unsigned*)(myws + OFF_TMO);

    STAMP_INIT();
    STAMP();
    if (tid == 0) {
        unsigned tk = __hip_atomic_fetch_add((unsigned*)(myws + OFF_CTR), 1u, __ATOMIC_RELAXED, __HIP_MEMORY_SCOPE_AGENT);
        *(LAS unsigned*)(lds + STAGE_BYTES) = tk / gridDim.x + 1u;
    }
    __syncthreads();
    const unsigned call = __builtin_amdgcn_readfirstlane(*(LAS unsigned*)(lds + STAGE_BYTES));
    const size_t par_off = OFF_STAGE + (size_t)(call & 1u) * PAR_BYTES;

    const int xcd = blockIdx.x & 7, idx = blockIdx.x >> 3;
    const int d = (rank + 1 + xcd) & 7;
    if (idx < 16) {
        const int wid = __builtin_amdgcn_readfirstlane(tid >> 6), lane = tid & 63;
        unsigned char* dws = p.ws[d];
        LAS unsigned char* wl = lds + wid * 4096;
        for (int u = idx * 8 + wid; u < NCH * 24 * 8; u += 128) {
            int j, tile;
            const int e = u & 7, uu = u >> 3;
            constexpr int NN = NCH - NDEF;
            if (uu < NN * 24) { j = uu / 24; tile = uu % 24; tile = tile < 8 ? 16 + tile : tile - 8; }
            else if (uu < NN * 24 + NDEF * 16) { const int v = uu - NN * 24; j = NN + (v >> 4); const int sl = v & 15; tile = sl < 8 ? 16 + sl : (sl - 8) * 2; }
            else { const int v = uu - (NN * 24 + NDEF * 16); j = NN + (v >> 3); tile = (v & 7) * 2 + 1; }
            const int t = j * NR + rank;
            size_t foff;
            if (tile < NMT) {
                const int mt = tile;
                unsigned char* dst = dws + par_off + ((size_t)(mt * NKT + t) * 2) * HTB + e * 4096 + lane * 16;
                f32x4 a[4][2];
#pragma unroll
                for (int i = 0; i < 4; ++i) {
                    const int b = e * 4096 + i * 1024 + lane * 16;
                    int R, C; stage_rc(b & (HTB - 1), R, C);
                    const float* src = p.x + (size_t)(mt * BM + (b >> 14) * HALF + R) * KS + j * BK + C;
                    a[i][0] = *(const f32x4*)src; a[i][1] = *(const f32x4*)(src + 4);
                }
#pragma unroll
                for (int i = 0; i < 4; ++i) st_wt(dst + i * 1024, cvt8(a[i][0], a[i][1]));
                foff = OFF_FX + ((size_t)((rank * NCH + j) * NMT + mt)) * 128 + e * 4;
            } else {
                const int nt = tile - NMT;
                const int k0 = (lane >> 3) * 8, n0 = (lane & 7) * 4;
                const float* src = p.w + (size_t)(j * BK + k0) * NTOT + d * NQ + nt * BM + e * 32 + n0;
                f32x4 a[8];
#pragma unroll
                for (int jj = 0; jj < 8; ++jj) a[jj] = *(const f32x4*)(src + (size_t)jj * NTOT);
#pragma unroll
                for (int i = 0; i < 4; ++i) {
                    h8 r;
#pragma unroll
                    for (int jj = 0; jj < 8; ++jj) r[jj] = (_Float16)a[jj][i];
                    *(LAS h8*)(wl + lds_byte(n0 + i, k0)) = r;
                }
                unsigned char* dst = dws + par_off + XA_BYTES + ((size_t)(nt * NKT + t) * 2) * HTB + e * 4096 + lane * 16;
                h8 o[4];
#pragma unroll
                for (int i = 0; i < 4; ++i) o[i] = *(LAS h8*)(wl + i * 1024 + lane * 16);
#pragma unroll
                for (int i = 0; i < 4; ++i) st_wt(dst + i * 1024, o[i]);
                foff = OFF_FW + ((size_t)((rank * NCH + j) * NNT + nt)) * 128 + e * 4;
            }
            asm volatile("s_waitcnt vmcnt(0)" ::: "memory");
            if (lane == 0) __hip_atomic_store((unsigned*)(dws + foff), call, __ATOMIC_RELAXED, __HIP_MEMORY_SCOPE_SYSTEM);
            STAMP();
        }
        if (blockIdx.x == 0 && tid < 64) {
            const unsigned* done = (const unsigned*)(myws + OFF_DONE);
            const unsigned long long t0 = __builtin_amdgcn_s_memrealtime();
            const unsigned want = call * 16u;
            bool bad = false;
            for (;;) {
                unsigned v0 = (lane >= NR || lane == rank) ? want : ld_sys(done + lane * 32);
                if (__all(v0 >= want)) break;
                if (__hip_atomic_load(tmo, __ATOMIC_RELAXED, __HIP_MEMORY_SCOPE_AGENT) == call ||
                    __builtin_amdgcn_s_memrealtime() - t0 > DEADLINE_TICKS) {
                    __hip_atomic_store(tmo, call, __ATOMIC_RELAXED, __HIP_MEMORY_SCOPE_AGENT);
                    bad = true; break;
                }
                __builtin_amdgcn_s_sleep(2);
            }
            __builtin_amdgcn_fence(__ATOMIC_ACQUIRE, "");
            if (bad && lane == 0) p.out[rank][0] = __builtin_nanf("");
            STAMP();
        }
        return;
    }

    const int mt = 2 * d + ((idx - 16) >> 3);
    const int nt = idx & 7;
    const int wid = __builtin_amdgcn_readfirstlane(tid >> 6), lane = tid & 63, wr = wid >> 2, wc = wid & 3, fr = lane & 15, fq = lane >> 4;
    const unsigned voff = (unsigned)tid * 16u;
    const unsigned ldsw = (unsigned)wid * 1024u;
    const int aoff = lds_byte(wr * 64 + fr, fq * 8), boff = lds_byte(wc * 32 + fr, fq * 8);
    const unsigned char* cA = myws + par_off + (size_t)mt * NKT * 2 * HTB;
    const unsigned char* cB = myws + par_off + XA_BYTES + (size_t)nt * NKT * 2 * HTB;
    const unsigned* fxp = (const unsigned*)(myws + OFF_FX) + (size_t)mt * 32;
    const unsigned* fwp = (const unsigned*)(myws + OFF_FW) + (size_t)nt * 32;

#define SA(b, h) (((b) * 2 + (h)) * HTB)
#define SB(b, h) ((4 + (b) * 2 + (h)) * HTB)
#define STAGE(bufoff, gbase) do { _Pragma("unroll") for (int _i = 0; _i < 2; ++_i) \
        __builtin_amdgcn_global_load_lds((const unsigned*)((gbase) + voff + _i * 8192), (LAS unsigned*)(lds + (bufoff) + ldsw + _i * 8192), 16, 0, 0); } while (0)
#define LDA(dst, b, h) do { _Pragma("unroll") for (int m = 0; m < 4; ++m) _Pragma("unroll") for (int k = 0; k < 2; ++k) dst[m][k] = *(const LAS h8*)(lds + SA(b, h) + aoff + m * 2048 + k * 1024); } while (0)
#define LDB(dst, b, h) do { _Pragma("unroll") for (int n = 0; n < 2; ++n) _Pragma("unroll") for (int k = 0; k < 2; ++k) dst[n][k] = *(const LAS h8*)(lds + SB(b, h) + boff + n * 2048 + k * 1024); } while (0)
#define MMA(ai, bj, At, Bt) do { __builtin_amdgcn_s_setprio(1); _Pragma("unroll") for (int m = 0; m < 4; ++m) _Pragma("unroll") for (int n = 0; n < 2; ++n) _Pragma("unroll") for (int k = 0; k < 2; ++k) \
        acc[ai][bj][m][n] = __builtin_amdgcn_mfma_f32_16x16x32_f16(Bt[n][k], At[m][k], acc[ai][bj][m][n], 0, 0, 0); __builtin_amdgcn_s_setprio(0); } while (0)
#define WAIT_V(n) asm volatile("s_waitcnt vmcnt(" #n ")" ::: "memory")
#define WAIT_L(n) asm volatile("s_waitcnt lgkmcnt(" #n ")" ::: "memory")
#define BAR __builtin_amdgcn_s_barrier()
#define SCHED __builtin_amdgcn_sched_barrier(0)
#define READY(j) do { \
        if (tid < 64) { \
            const unsigned* fp = ((lane & 8) == 0 ? fxp + (size_t)(((lane & 7) * NCH + (j)) * NMT) * 32 : fwp + (size_t)(((lane & 7) * NCH + (j)) * NNT) * 32) + (lane >> 4) * 2; \
            const unsigned long long t0 = __builtin_amdgcn_s_memrealtime(); \
            for (;;) { \
                const unsigned long long v = __hip_atomic_load((const unsigned long long*)fp, __ATOMIC_RELAXED, __HIP_MEMORY_SCOPE_SYSTEM); \
                if (__all((unsigned)v >= call && (unsigned)(v >> 32) >= call)) break; \
                if (__hip_atomic_load(tmo, __ATOMIC_RELAXED, __HIP_MEMORY_SCOPE_AGENT) == call || \
                    __builtin_amdgcn_s_memrealtime() - t0 > DEADLINE_TICKS) { \
                    __hip_atomic_store(tmo, call, __ATOMIC_RELAXED, __HIP_MEMORY_SCOPE_AGENT); break; } \
                __builtin_amdgcn_s_sleep(2); \
            } \
            __builtin_amdgcn_fence(__ATOMIC_ACQUIRE, ""); \
            asm volatile("s_waitcnt vmcnt(0)" ::: "memory"); \
        } \
        STAMP(); \
        asm volatile("" ::: "memory"); __builtin_amdgcn_s_barrier(); asm volatile("" ::: "memory"); \
    } while (0)

    f32x4 acc[2][2][4][2];
#pragma unroll
    for (int a = 0; a < 2; ++a)
#pragma unroll
        for (int b = 0; b < 2; ++b)
#pragma unroll
            for (int m = 0; m < 4; ++m)
#pragma unroll
                for (int n = 0; n < 2; ++n) acc[a][b][m][n] = (f32x4){0.f, 0.f, 0.f, 0.f};
    h8 At[4][2], B0[2][2], B1[2][2];
    constexpr size_t kstep = 2 * HTB, hstep = HTB;

    READY(0);
    STAGE(SB(0, 0), cB); STAGE(SA(0, 0), cA); STAGE(SB(0, 1), cB + hstep); STAGE(SA(0, 1), cA + hstep);
    if (wr == 1) BAR;
    WAIT_V(4); BAR;
    STAGE(SB(1, 0), cB + kstep); STAGE(SA(1, 0), cA + kstep); STAGE(SB(1, 1), cB + hstep + kstep);
    WAIT_V(6); BAR;
    for (int t = 0; t < NKT; t += 2) {
        const bool last = (t == NKT - 2);
        const unsigned char* a1 = cA + (size_t)(t + 1) * kstep;
        const unsigned char* a2 = last ? cA : cA + (size_t)(t + 2) * kstep;
        const unsigned char* b2 = last ? cB : cB + (size_t)(t + 2) * kstep;
        const unsigned char* a3 = a2 + kstep; const unsigned char* b3 = b2 + kstep;
        if (!last && ((t + 2) & 7) == 0) READY((t + 2) >> 3);
        LDB(B0, 0, 0); SCHED; LDA(At, 0, 0); STAGE(SA(1, 1), a1 + hstep);
        WAIT_L(8); BAR; WAIT_L(0); MMA(0, 0, At, B0); BAR; SCHED;
        LDB(B1, 0, 1); STAGE(SB(0, 0), b2);
        BAR; WAIT_L(0); MMA(0, 1, At, B1); BAR;
        LDA(At, 0, 1); STAGE(SA(0, 0), a2);
        BAR; WAIT_L(0); MMA(1, 0, At, B0); BAR; SCHED;
        STAGE(SB(0, 1), b2 + hstep);
        WAIT_V(6); BAR; MMA(1, 1, At, B1); BAR;
        LDB(B0, 1, 0); SCHED; LDA(At, 1, 0); STAGE(SA(0, 1), a2 + hstep);
        WAIT_L(8); BAR; WAIT_L(0); MMA(0, 0, At, B0); BAR; SCHED;
        LDB(B1, 1, 1); STAGE(SB(1, 0), b3);
        BAR; WAIT_L(0); MMA(0, 1, At, B1); BAR;
        LDA(At, 1, 1); STAGE(SA(1, 0), a3);
        BAR; WAIT_L(0); MMA(1, 0, At, B0); BAR; SCHED;
        STAGE(SB(1, 1), b3 + hstep);
        WAIT_V(6); BAR; MMA(1, 1, At, B1); BAR;
    }
    WAIT_V(0);
    if (wr == 0) BAR;
    BAR;

    STAMP();
    const bool bad = (__hip_atomic_load(tmo, __ATOMIC_RELAXED, __HIP_MEMORY_SCOPE_AGENT) == call);
    float* obase = p.out[d] + (size_t)((mt & 1) * BM + wr * 64 + fr) * NTOT + rank * NQ + nt * BM + wc * 32 + 4 * fq;
    const float qn = __builtin_nanf("");
#pragma unroll
    for (int ai = 0; ai < 2; ++ai)
#pragma unroll
        for (int m = 0; m < 4; ++m) {
            float* rowp = obase + (size_t)(ai * HALF + m * 16) * NTOT;
#pragma unroll
            for (int bj = 0; bj < 2; ++bj)
#pragma unroll
                for (int n = 0; n < 2; ++n) {
                    f32x4 v = acc[ai][bj][m][n];
                    v[0] = v[0] > 0.f ? v[0] : 0.f; v[1] = v[1] > 0.f ? v[1] : 0.f;
                    v[2] = v[2] > 0.f ? v[2] : 0.f; v[3] = v[3] > 0.f ? v[3] : 0.f;
                    if (bad) v = (f32x4){qn, qn, qn, qn};
                    st_wt(rowp + bj * HALF + n * 16, v);
                }
        }
    STAMP();
    asm volatile("s_waitcnt vmcnt(0)" ::: "memory");
    __syncthreads();
    if (tid == 0 && d != rank)
        __hip_atomic_fetch_add((unsigned*)(p.ws[d] + OFF_DONE + (size_t)rank * 128), 1u, __ATOMIC_RELAXED, __HIP_MEMORY_SCOPE_SYSTEM);
    STAMP();
}

}

extern "C" void kernel_launch(int rank, int nranks, ncclComm_t comm,
                              hipStream_t stream,
                              void* const* d_in, const int* in_sizes, int n_in,
                              void* d_out, int out_size,
                              const int* in_dtypes, int out_dtype,
                              void* const* d_out_all,
                              void* const* d_ws_all, size_t ws_bytes) {
    if (nranks != NR || ws_bytes < WS_NEED) { fprintf(stderr, "unsupported configuration\n"); return; }
    static bool init = false;
    if (!init) {
        HIP_CHECK(hipFuncSetAttribute((const void*)fused_kernel, hipFuncAttributeMaxDynamicSharedMemorySize, LDS_BYTES));
        init = true;
    }
    Params p;
    p.x = (const float*)d_in[0];
    p.w = (const float*)d_in[1];
    for (int i = 0; i < NR; ++i) { p.out[i] = (float*)d_out_all[i]; p.ws[i] = (unsigned char*)d_ws_all[i]; }
    p.rank = rank;
    fused_kernel<<<256, 512, LDS_BYTES, stream>>>(p);
    HIP_CHECK(hipGetLastError());
}
```

```cpp
#include <hip/hip_runtime.h>
#include <rccl/rccl.h>
#include <cstdio>

#define HIP_CHECK(expr) do { hipError_t _e = (expr); if (_e != hipSuccess) { \
    fprintf(stderr, "HIP error %s at %s:%d\n", hipGetErrorString(_e), __FILE__, __LINE__); \
    return; } } while (0)

namespace {

#define LAS __attribute__((address_space(3)))
typedef _Float16 h8 __attribute__((ext_vector_type(8)));
typedef float f32x4 __attribute__((ext_vector_type(4)));
typedef unsigned u32x4 __attribute__((ext_vector_type(4)));

constexpr int NR = 8;
constexpr int MTOT = 4096;
constexpr int KS = 512;
constexpr int NTOT = 16384;
constexpr int NQ = NTOT / NR;
constexpr int BM = 256, BK = 64, HALF = 128;
constexpr int HTB = HALF * BK * 2;
constexpr int NMT = MTOT / BM;
constexpr int NNT = NQ / BM;
constexpr int NCH = KS / BK;
constexpr int NKT = NR * NCH;
constexpr int OBATCH = 512 + 16384;
constexpr int OSLOT = 2 * OBATCH;
constexpr int LDS_BYTES = 136 * 1024;
constexpr int CALLW = LDS_BYTES - 64;
static_assert(8 * (OBATCH + 256) <= CALLW && 8 * HTB <= CALLW, "lds");
#define OTHR 3
#define ROT 4
#define OTAIL 5
#define NDEF 6

constexpr size_t OFF_CTR = 0;
constexpr size_t OFF_TMO = 256;
constexpr size_t OFF_Q = 1024;
constexpr int NPIECE = NCH * 24 * 8;
constexpr size_t OFF_FX = 4096;
constexpr size_t OFF_FW = OFF_FX + (size_t)NR * NCH * NMT * 128;
constexpr size_t OFF_DONE = OFF_FW + (size_t)NR * NCH * NNT * 128;
constexpr size_t OFF_STAGE = 1ull << 20;
constexpr size_t XA_BYTES = (size_t)NMT * NKT * 2 * HTB;
constexpr size_t WB_BYTES = (size_t)NNT * NKT * 2 * HTB;
constexpr size_t PAR_BYTES = XA_BYTES + WB_BYTES;
constexpr size_t OFF_OUT = OFF_STAGE + 2 * PAR_BYTES;
constexpr size_t OFF_PX = OFF_OUT + (size_t)NR * 128 * OSLOT + 4096;
constexpr size_t OFF_PXF = 384 * 1024;
constexpr size_t WS_NEED = OFF_PX + (size_t)64 * 2 * 8 * 16384;
static_assert(OFF_DONE + (size_t)NR * 128 * 64 <= OFF_PXF && OFF_PXF + 1024 * 64 <= 512 * 1024, "flag region");

#define STAMP() do {} while (0)
#define STAMP_INIT()
#define STAMP_BEGIN() do {} while (0)
#define STAMP_END() do {} while (0)
constexpr unsigned long long DEADLINE_TICKS = 40000000ull;

struct Params {
    const float* x;
    const float* w;
    float* out[NR];
    unsigned char* ws[NR];
    int rank;
};

__device__ __forceinline__ int lds_byte(int r, int c) {
    const int st = (r >> 4) * 2 + (c >> 5), rr = r & 15, cc = c & 31, ob = rr * 64 + cc * 2;
    return st * 1024 + (ob ^ (((ob >> 9) & 1) << 5));
}
__device__ __forceinline__ void stage_rc(int b, int& R, int& C) {
    const int st = b / 1024, sb = b % 1024, swz = sb ^ (((sb >> 9) & 1) << 5);
    R = (st >> 1) * 16 + swz / 64; C = (st & 1) * 32 + (swz % 64) / 2;
}
__device__ __forceinline__ unsigned ld_sys(const unsigned* p) {
    return __hip_atomic_load(p, __ATOMIC_RELAXED, __HIP_MEMORY_SCOPE_SYSTEM);
}
__device__ __forceinline__ h8 cvt8(const f32x4 a, const f32x4 b) {
    h8 r;
    r[0] = (_Float16)a[0]; r[1] = (_Float16)a[1]; r[2] = (_Float16)a[2]; r[3] = (_Float16)a[3];
    r[4] = (_Float16)b[0]; r[5] = (_Float16)b[1]; r[6] = (_Float16)b[2]; r[7] = (_Float16)b[3];
    return r;
}
template <class T> __device__ __forceinline__ void st_wt(void* ptr, const T v) {
    asm volatile("global_store_dwordx4 %0, %1, off sc0 sc1\n\ts_nop 1" :: "v"(ptr), "v"(v) : "memory");
}
#define WAIT_V(n) asm volatile("s_waitcnt vmcnt(" #n ")" ::: "memory")
#define WAIT_L(n) asm volatile("s_waitcnt lgkmcnt(" #n ")" ::: "memory")
#define BAR __builtin_amdgcn_s_barrier()
#define SCHED __builtin_amdgcn_sched_barrier(0)

__device__ __forceinline__ unsigned wave_incl_scan(unsigned x, int lane) {
#pragma unroll
    for (int o = 1; o < 64; o <<= 1) { const unsigned y = __shfl_up(x, o, 64); if (lane >= o) x += y; }
    return x;
}

__device__ __forceinline__ void wait_round(const unsigned* fxp, const unsigned* fwp, int j, unsigned call, unsigned* tmo, int lane) {
    const unsigned* fp = ((lane & 8) == 0 ? fxp + (size_t)(((lane & 7) * NCH + j) * NMT) * 32 : fwp + (size_t)(((lane & 7) * NCH + j) * NNT) * 32) + (lane >> 4) * 2;
    const unsigned long long t0 = __builtin_amdgcn_s_memrealtime();
    for (;;) {
        const unsigned long long v = __hip_atomic_load((const unsigned long long*)fp, __ATOMIC_RELAXED, __HIP_MEMORY_SCOPE_SYSTEM);
        if (__all((unsigned)v >= call && (unsigned)(v >> 32) >= call)) break;
        if (__hip_atomic_load(tmo, __ATOMIC_RELAXED, __HIP_MEMORY_SCOPE_AGENT) == call ||
            __builtin_amdgcn_s_memrealtime() - t0 > DEADLINE_TICKS) {
            __hip_atomic_store(tmo, call, __ATOMIC_RELAXED, __HIP_MEMORY_SCOPE_AGENT); break; }
        __builtin_amdgcn_s_sleep(2);
    }
    __builtin_amdgcn_fence(__ATOMIC_ACQUIRE, "");
    asm volatile("s_waitcnt vmcnt(0)" ::: "memory");
}

__device__ __forceinline__ void store_dense(const f32x4 (&a)[2][4][2], float* obase, bool bad) {
    const float qn = __builtin_nanf("");
#pragma unroll
    for (int m = 0; m < 4; ++m)
#pragma unroll
        for (int bj = 0; bj < 2; ++bj)
#pragma unroll
            for (int n = 0; n < 2; ++n) {
                f32x4 v = a[bj][m][n];
#pragma unroll
                for (int e = 0; e < 4; ++e) v[e] = v[e] <= 0.f ? 0.f : v[e];
                if (bad) v = (f32x4){qn, qn, qn, qn};
                st_wt(obase + (size_t)(m * 16) * NTOT + bj * HALF + n * 16, v);
            }
}

__device__ __forceinline__ unsigned pack_sparse(const f32x4 (&a)[2][4][2], LAS unsigned char* wb, int lane, bool bad) {
    if (bad) {
        for (int i = lane; i < OBATCH / 4; i += 64) *(LAS unsigned*)(wb + i * 4) = 0xffffffffu;
        return 4096u;
    }
    const unsigned dummy = (unsigned)(OBATCH - 512) / 4u + (unsigned)lane;
    unsigned mw[2];
#pragma unroll
    for (int h = 0; h < 2; ++h) {
        unsigned mm = 0;
#pragma unroll
        for (int i = 31; i >= 0; --i) { const int f = h * 8 + (i >> 2); mm = mm + mm + ((a[f >> 3][(f >> 1) & 3][f & 1][i & 3] <= 0.f) ? 0u : 1u); }
        mw[h] = mm;
    }
    const unsigned cnt = __builtin_popcount(mw[0]) + __builtin_popcount(mw[1]);
    const unsigned inc = wave_incl_scan(cnt, lane);
    const unsigned tot = __builtin_amdgcn_readlane(inc, 63);
    *(LAS unsigned long long*)(wb + lane * 8) = ((unsigned long long)mw[1] << 32) | mw[0];
    unsigned pos = inc - cnt;
#pragma unroll
    for (int f = 0; f < 16; ++f) {
#pragma unroll
        for (int e = 0; e < 4; ++e) {
            const float x = a[f >> 3][(f >> 1) & 3][f & 1][e];
            const bool z = x <= 0.f;
            *(LAS float*)(wb + 512 + (z ? dummy : pos) * 4) = x;
            pos += z ? 0u : 1u;
        }
        if (f & 1) SCHED;
    }
    return tot;
}
__device__ __forceinline__ void flush_sparse(LAS unsigned char* wb, unsigned char* dst, int lane, unsigned tot) {
    const unsigned n16 = (512u + tot * 4u + 15u) >> 4;
    for (unsigned i = lane; i < n16; i += 64) {
        st_wt(dst + i * 16, *(LAS f32x4*)(wb + i * 16));
        if (i + OTAIL * 64 < n16) asm volatile("s_waitcnt vmcnt(%0)" :: "i"(OTHR) : "memory");
    }
}
__device__ __forceinline__ void flag_sparse(unsigned long long* flag, int lane, unsigned call, unsigned tot) {
    WAIT_V(0);
    if (lane == 0) __hip_atomic_store(flag, ((unsigned long long)tot << 32) | call, __ATOMIC_RELAXED, __HIP_MEMORY_SCOPE_SYSTEM);
}

__device__ __forceinline__ void recv_batch(unsigned char* myws, float* out, int src, int tl, int ai, int wid, int lane,
                                           LAS unsigned char* wl, unsigned call, unsigned* tmo) {
    const int wr = wid >> 2, wc = wid & 3, fr = lane & 15, fq = lane >> 4;
    const int sidx = (src * 16 + tl) * 8 + wid;
    const unsigned long long* fl = (const unsigned long long*)(myws + OFF_DONE + (size_t)sidx * 64) + ai;
    const unsigned char* sp = myws + OFF_OUT + (size_t)sidx * OSLOT + ai * OBATCH;
    float* obase = out + (size_t)((tl >> 3) * BM + ai * HALF + wr * 64 + fr) * NTOT + src * NQ + (tl & 7) * BM + wc * 32 + 4 * fq;
    const unsigned long long t0 = __builtin_amdgcn_s_memrealtime();
    bool bad = false;
    unsigned tot = 0;
    for (;;) {
        const unsigned long long v = __hip_atomic_load(fl, __ATOMIC_RELAXED, __HIP_MEMORY_SCOPE_SYSTEM);
        tot = __builtin_amdgcn_readfirstlane((unsigned)(v >> 32));
        if ((unsigned)__builtin_amdgcn_readfirstlane((unsigned)v) >= call) break;
        if (__hip_atomic_load(tmo, __ATOMIC_RELAXED, __HIP_MEMORY_SCOPE_AGENT) == call ||
            __builtin_amdgcn_s_memrealtime() - t0 > DEADLINE_TICKS) {
            __hip_atomic_store(tmo, call, __ATOMIC_RELAXED, __HIP_MEMORY_SCOPE_AGENT);
            bad = true; tot = 0; break;
        }
        __builtin_amdgcn_s_sleep(4);
    }
    tot = tot > 4096u ? 4096u : tot;
    __builtin_amdgcn_fence(__ATOMIC_ACQUIRE, "wavefront");
    const unsigned nit = (tot * 4u + 1023u) >> 10;
    for (unsigned i = 0; i < nit; ++i)
        __builtin_amdgcn_global_load_lds((const unsigned*)(sp + 512 + i * 1024 + lane * 16), (LAS unsigned*)(wl + i * 1024), 16, 0, 17);
    const unsigned long long mk = bad ? 0ull : __hip_atomic_load((const unsigned long long*)(sp + lane * 8), __ATOMIC_RELAXED, __HIP_MEMORY_SCOPE_SYSTEM);
    const unsigned mlo = (unsigned)mk, mhi = (unsigned)(mk >> 32);
    const unsigned cnt = __builtin_popcount(mlo) + __builtin_popcount(mhi);
    const unsigned inc = wave_incl_scan(cnt, lane);
    WAIT_V(0);
    unsigned pos = inc - cnt;
    f32x4 v[16];
#pragma unroll
    for (int f = 0; f < 16; ++f)
#pragma unroll
        for (int e = 0; e < 4; ++e) {
            const unsigned k = ((f < 8 ? mlo : mhi) >> ((f & 7) * 4 + e)) & 1u;
            const float x = *(LAS float*)(wl + pos * 4);
            v[f][e] = k ? x : 0.f;
            pos += k;
        }
    const float qn = __builtin_nanf("");
#pragma unroll
    for (int f = 0; f < 16; ++f) {
        if (bad) v[f] = (f32x4){qn, qn, qn, qn};
        *(f32x4*)(obase + (size_t)(((f >> 1) & 3) * 16) * NTOT + (f >> 3) * HALF + (f & 1) * 16) = v[f];
    }
}

__global__ void __launch_bounds__(512, 2) fused_kernel(Params p) {
    extern __shared__ __attribute__((aligned(16))) unsigned char smem[];
    LAS unsigned char* lds = (LAS unsigned char*)smem;
    const int tid = threadIdx.x;
    const int rank = p.rank;
    unsigned char* myws = p.ws[rank];
    unsigned* tmo = (unsigned*)(myws + OFF_TMO);

    STAMP_INIT();
    STAMP();
    if (tid == 0) {
        unsigned tk = __hip_atomic_fetch_add((unsigned*)(myws + OFF_CTR), 1u, __ATOMIC_RELAXED, __HIP_MEMORY_SCOPE_AGENT);
        *(LAS unsigned*)(lds + CALLW) = tk / gridDim.x + 1u;
    }
    __syncthreads();
    const unsigned call = __builtin_amdgcn_readfirstlane(*(LAS unsigned*)(lds + CALLW));
    const size_t par_off = OFF_STAGE + (size_t)(call & 1u) * PAR_BYTES;
    STAMP_BEGIN();

    const int xcd = blockIdx.x & 7, idx = blockIdx.x >> 3;
    const int d = (rank + 1 + ((xcd + ROT) & 7)) & 7;
    const int wid = __builtin_amdgcn_readfirstlane(tid >> 6), lane = tid & 63, wr = wid >> 2, wc = wid & 3, fr = lane & 15, fq = lane >> 4;
    if (idx < 8) {
        unsigned char* dws = p.ws[d];
        LAS unsigned char* wl = lds + wid * OBATCH;
        unsigned* qctr = (unsigned*)(myws + OFF_Q + xcd * 128);
        const unsigned qbase = (call - 1u) * (unsigned)NPIECE;
        auto piece_of = [&](int u, int& j, int& tile) {
            const int uu = u >> 3;
            constexpr int NN = NCH - NDEF;
            if (uu < NN * 24) { j = uu / 24; tile = uu % 24; tile = tile < 8 ? 16 + tile : tile - 8; }
            else if (uu < NN * 24 + NDEF * 16) { const int v = uu - NN * 24; j = NN + (v >> 4); const int sl = v & 15; tile = sl < 8 ? 16 + sl : (sl - 8) * 2; }
            else { const int v = uu - (NN * 24 + NDEF * 16); j = NN + (v >> 3); tile = (v & 7) * 2 + 1; }
        };
        f32x4 a[8];
        unsigned tk;
        const unsigned one = 1u;
        auto load_piece = [&](int u) {
            int j, tile; piece_of(u, j, tile);
            const int e = u & 7;
            const float* sp[8];
            if (tile < NMT) {
#pragma unroll
                for (int i = 0; i < 4; ++i) {
                    const int b = e * 4096 + i * 1024 + lane * 16;
                    int R, C; stage_rc(b & (HTB - 1), R, C);
                    sp[2 * i] = p.x + (size_t)(tile * BM + (b >> 14) * HALF + R) * KS + j * BK + C;
                    sp[2 * i + 1] = sp[2 * i] + 4;
                }
            } else {
                const float* src = p.w + (size_t)(j * BK + (lane >> 3) * 8) * NTOT + d * NQ + (tile - NMT) * BM + e * 32 + (lane & 7) * 4;
#pragma unroll
                for (int jj = 0; jj < 8; ++jj) sp[jj] = src + (size_t)jj * NTOT;
            }
            unsigned long long sv;
            asm volatile(
                "global_load_dwordx4 %0, %10, off\n\tglobal_load_dwordx4 %1, %11, off\n\tglobal_load_dwordx4 %2, %12, off\n\tglobal_load_dwordx4 %3, %13, off\n\t"
                "global_load_dwordx4 %4, %14, off\n\tglobal_load_dwordx4 %5, %15, off\n\tglobal_load_dwordx4 %6, %16, off\n\tglobal_load_dwordx4 %7, %17, off\n\t"
                "s_mov_b64 %9, exec\n\ts_mov_b64 exec, 1\n\tglobal_atomic_add %8, %18, %19, off sc0\n\ts_mov_b64 exec, %9"
                : "=&v"(a[0]), "=&v"(a[1]), "=&v"(a[2]), "=&v"(a[3]), "=&v"(a[4]), "=&v"(a[5]), "=&v"(a[6]), "=&v"(a[7]), "=&v"(tk), "=&s"(sv)
                : "v"(sp[0]), "v"(sp[1]), "v"(sp[2]), "v"(sp[3]), "v"(sp[4]), "v"(sp[5]), "v"(sp[6]), "v"(sp[7]), "v"(qctr), "v"(one) : "memory");
        };
#define LANDED(n) asm volatile("s_waitcnt vmcnt(" #n ")" : "+v"(a[0]), "+v"(a[1]), "+v"(a[2]), "+v"(a[3]), "+v"(a[4]), "+v"(a[5]), "+v"(a[6]), "+v"(a[7]), "+v"(tk) :: "memory")
        int u = idx * 8 + wid;
        load_piece(u);
        LANDED(0);
        while (u < NPIECE) {
            const unsigned nxt = (unsigned)__builtin_amdgcn_readfirstlane(tk) - qbase + 64u;
            int j, tile; piece_of(u, j, tile);
            const int e = u & 7, t = j * NR + rank;
            size_t foff;
            if (tile < NMT) {
                unsigned char* dst = dws + par_off + ((size_t)(tile * NKT + t) * 2) * HTB + e * 4096 + lane * 16;
#pragma unroll
                for (int i = 0; i < 4; ++i) st_wt(dst + i * 1024, cvt8(a[2 * i], a[2 * i + 1]));
                foff = OFF_FX + ((size_t)((rank * NCH + j) * NMT + tile)) * 128 + e * 4;
            } else {
                const int nt = tile - NMT, k0 = (lane >> 3) * 8, n0 = (lane & 7) * 4;
#pragma unroll
                for (int i = 0; i < 4; ++i) {
                    h8 r;
#pragma unroll
                    for (int jj = 0; jj < 8; ++jj) r[jj] = (_Float16)a[jj][i];
                    *(LAS h8*)(wl + lds_byte(n0 + i, k0)) = r;
                }
                unsigned char* dst = dws + par_off + XA_BYTES + ((size_t)(nt * NKT + t) * 2) * HTB + e * 4096 + lane * 16;
                h8 o[4];
#pragma unroll
                for (int i = 0; i < 4; ++i) o[i] = *(LAS h8*)(wl + i * 1024 + lane * 16);
#pragma unroll
                for (int i = 0; i < 4; ++i) st_wt(dst + i * 1024, o[i]);
                foff = OFF_FW + ((size_t)((rank * NCH + j) * NNT + nt)) * 128 + e * 4;
            }
            u = (int)nxt;
            if (u < NPIECE) {
                load_piece(u);
                WAIT_V(9);
            } else {
                if (lane == 0) __hip_atomic_fetch_add(qctr + 512, 1u, __ATOMIC_RELAXED, __HIP_MEMORY_SCOPE_AGENT);
                WAIT_V(0);
            }
            if (lane == 0) __hip_atomic_store((unsigned*)(dws + foff), call, __ATOMIC_RELAXED, __HIP_MEMORY_SCOPE_SYSTEM);
            LANDED(1);
            STAMP();
        }
        if (d == rank) return;
        for (int bt = 0; bt < 3; ++bt) {
            recv_batch(myws, p.out[rank], d, idx + (bt >> 1) * 8, bt & 1, wid, lane, wl, call, tmo);
            STAMP();
        }
        STAMP_END();
        return;
    }

    const bool full = idx < 16;
    const int mt = 2 * d + (full ? 0 : 1);
    const int nt = idx & 7;
    const int hh = (idx >> 3) & 1;
    const unsigned voff = (unsigned)tid * 16u;
    const unsigned ldsw = (unsigned)wid * 1024u;
    const int aoff = lds_byte(wr * 64 + fr, fq * 8), boff = lds_byte(wc * 32 + fr, fq * 8);
    const unsigned char* cA = myws + par_off + (size_t)mt * NKT * 2 * HTB;
    const unsigned char* cB = myws + par_off + XA_BYTES + (size_t)nt * NKT * 2 * HTB;
    const unsigned* fxp = (const unsigned*)(myws + OFF_FX) + (size_t)mt * 32;
    const unsigned* fwp = (const unsigned*)(myws + OFF_FW) + (size_t)nt * 32;
    constexpr size_t kstep = 2 * HTB, hstep = HTB;
    const int sidx = (rank * 16 + (mt & 1) * 8 + nt) * 8 + wid;
    unsigned char* slot = p.ws[d] + OFF_OUT + (size_t)sidx * OSLOT;
    unsigned long long* oflag = (unsigned long long*)(p.ws[d] + OFF_DONE + (size_t)sidx * 64);
    float* obase = p.out[d] + (size_t)((mt & 1) * BM + wr * 64 + fr) * NTOT + rank * NQ + nt * BM + wc * 32 + 4 * fq;
    LAS unsigned char* wb = lds + wid * (OBATCH + 256);

#define STAGE(bufoff, gbase) do { _Pragma("unroll") for (int _i = 0; _i < 2; ++_i) \
        __builtin_amdgcn_global_load_lds((const unsigned*)((gbase) + voff + _i * 8192), (LAS unsigned*)(lds + (bufoff) + ldsw + _i * 8192), 16, 0, 0); } while (0)
#define LDFA(dst, off) do { _Pragma("unroll") for (int m = 0; m < 4; ++m) _Pragma("unroll") for (int k = 0; k < 2; ++k) dst[m][k] = *(const LAS h8*)(lds + (off) + aoff + m * 2048 + k * 1024); } while (0)
#define LDFB(dst, off) do { _Pragma("unroll") for (int n = 0; n < 2; ++n) _Pragma("unroll") for (int k = 0; k < 2; ++k) dst[n][k] = *(const LAS h8*)(lds + (off) + boff + n * 2048 + k * 1024); } while (0)
#define MMAX(C, At, Bt) do { _Pragma("unroll") for (int m = 0; m < 4; ++m) _Pragma("unroll") for (int n = 0; n < 2; ++n) _Pragma("unroll") for (int k = 0; k < 2; ++k) \
        C[m][n] = __builtin_amdgcn_mfma_f32_16x16x32_f16(Bt[n][k], At[m][k], C[m][n], 0, 0, 0); } while (0)
#define READY(j) do { if (tid < 64) wait_round(fxp, fwp, (j), call, tmo, lane); STAMP(); \
        asm volatile("" ::: "memory"); __builtin_amdgcn_s_barrier(); asm volatile("" ::: "memory"); } while (0)

#define SA(b, h) (((b) * 2 + (h)) * HTB)
#define SB(b, h) ((4 + (b) * 2 + (h)) * HTB)
#define MMA(ai, bj, At, Bt) do { __builtin_amdgcn_s_setprio(1); MMAX(acc[ai][bj], At, Bt); __builtin_amdgcn_s_setprio(0); } while (0)
    f32x4 acc[2][2][4][2];
#pragma unroll
    for (int a = 0; a < 2; ++a)
#pragma unroll
        for (int b = 0; b < 2; ++b)
#pragma unroll
            for (int m = 0; m < 4; ++m)
#pragma unroll
                for (int n = 0; n < 2; ++n) acc[a][b][m][n] = (f32x4){0.f, 0.f, 0.f, 0.f};
    h8 At[4][2], B0[2][2], B1[2][2];
    const int nti = full ? NKT : NKT / 2;
    const size_t tstep = full ? kstep : 2 * kstep;
    const int rsh = full ? 3 : 2;
    if (!full) { cA += (size_t)hh * kstep; cB += (size_t)hh * kstep; }
    READY(0);
    STAGE(SB(0, 0), cB); STAGE(SA(0, 0), cA); STAGE(SB(0, 1), cB + hstep); STAGE(SA(0, 1), cA + hstep);
    if (wr == 1) BAR;
    WAIT_V(4); BAR;
    STAGE(SB(1, 0), cB + tstep); STAGE(SA(1, 0), cA + tstep); STAGE(SB(1, 1), cB + hstep + tstep);
    WAIT_V(6); BAR;
    for (int t = 0; t < nti; t += 2) {
        const bool last = (t == nti - 2);
        const unsigned char* a1 = cA + (size_t)(t + 1) * tstep;
        const unsigned char* a2 = last ? cA : cA + (size_t)(t + 2) * tstep;
        const unsigned char* b2 = last ? cB : cB + (size_t)(t + 2) * tstep;
        const unsigned char* a3 = a2 + tstep; const unsigned char* b3 = b2 + tstep;
        if (!last && ((t + 2) & ((1 << rsh) - 1)) == 0) READY((t + 2) >> rsh);
        LDFB(B0, SB(0, 0)); SCHED; LDFA(At, SA(0, 0)); STAGE(SA(1, 1), a1 + hstep);
        WAIT_L(8); BAR; WAIT_L(0); MMA(0, 0, At, B0); BAR; SCHED;
        LDFB(B1, SB(0, 1)); STAGE(SB(0, 0), b2);
        BAR; WAIT_L(0); MMA(0, 1, At, B1); BAR;
        LDFA(At, SA(0, 1)); STAGE(SA(0, 0), a2);
        BAR; WAIT_L(0); MMA(1, 0, At, B0); BAR; SCHED;
        STAGE(SB(0, 1), b2 + hstep);
        WAIT_V(6); BAR; MMA(1, 1, At, B1); BAR;
        LDFB(B0, SB(1, 0)); SCHED; LDFA(At, SA(1, 0)); STAGE(SA(0, 1), a2 + hstep);
        WAIT_L(8); BAR; WAIT_L(0); MMA(0, 0, At, B0); BAR; SCHED;
        LDFB(B1, SB(1, 1)); STAGE(SB(1, 0), b3);
        BAR; WAIT_L(0); MMA(0, 1, At, B1); BAR;
        LDFA(At, SA(1, 1)); STAGE(SA(1, 0), a3);
        BAR; WAIT_L(0); MMA(1, 0, At, B0); BAR; SCHED;
        STAGE(SB(1, 1), b3 + hstep);
        WAIT_V(6); BAR; MMA(1, 1, At, B1); BAR;
    }
    WAIT_V(0);
    if (wr == 0) BAR;
    BAR;
    STAMP();
    if (full) {
        const bool bad = (__hip_atomic_load(tmo, __ATOMIC_RELAXED, __HIP_MEMORY_SCOPE_AGENT) == call);
        if (d == rank) {
            store_dense(acc[0], obase, bad);
            store_dense(acc[1], obase + (size_t)HALF * NTOT, bad);
        } else {
            const unsigned tot0 = pack_sparse(acc[0], wb, lane, bad);
            if (wid == 0) {
                const unsigned* qctr = (const unsigned*)(myws + OFF_Q + xcd * 128) + 512;
                const unsigned long long t0 = __builtin_amdgcn_s_memrealtime();
                while ((unsigned)__builtin_amdgcn_readfirstlane(__hip_atomic_load(qctr, __ATOMIC_RELAXED, __HIP_MEMORY_SCOPE_AGENT)) < call * 64u &&
                       __builtin_amdgcn_s_memrealtime() - t0 < DEADLINE_TICKS)
                    __builtin_amdgcn_s_sleep(8);
            }
            asm volatile("" ::: "memory"); BAR; asm volatile("" ::: "memory");
            flush_sparse(wb, slot, lane, tot0);
            const unsigned tot1 = pack_sparse(acc[1], wb, lane, bad);
            flag_sparse(oflag, lane, call, tot0);
            flush_sparse(wb, slot + OBATCH, lane, tot1);
            STAMP();
            flag_sparse(oflag + 1, lane, call, tot1);
            STAMP();
            recv_batch(myws, p.out[rank], d, idx, 1, wid, lane, wb, call, tmo);
        }
    } else {
        const int px = (xcd * 8 + nt) * 2;
        unsigned char* mine = myws + OFF_PX + (size_t)((px + hh) * 8 + wid) * 16384 + lane * 16;
        const unsigned char* theirs = myws + OFF_PX + (size_t)((px + (hh ^ 1)) * 8 + wid) * 16384 + lane * 16;
        unsigned* pfl = (unsigned*)(myws + OFF_PXF);
#pragma unroll
        for (int f = 0; f < 16; ++f) {
            const f32x4 lo = acc[0][f >> 3][(f >> 1) & 3][f & 1], hi = acc[1][f >> 3][(f >> 1) & 3][f & 1];
            st_wt(mine + f * 1024, hh ? lo : hi);
            acc[0][f >> 3][(f >> 1) & 3][f & 1] = hh ? hi : lo;
        }
        WAIT_V(0);
        if (lane == 0) __hip_atomic_store(pfl + ((px + hh) * 8 + wid) * 16, call, __ATOMIC_RELAXED, __HIP_MEMORY_SCOPE_SYSTEM);
        {
            const unsigned* fl = pfl + ((px + (hh ^ 1)) * 8 + wid) * 16;
            const unsigned long long t0 = __builtin_amdgcn_s_memrealtime();
            while ((unsigned)__builtin_amdgcn_readfirstlane(ld_sys(fl)) < call) {
                if (__hip_atomic_load(tmo, __ATOMIC_RELAXED, __HIP_MEMORY_SCOPE_AGENT) == call ||
                    __builtin_amdgcn_s_memrealtime() - t0 > DEADLINE_TICKS) {
                    __hip_atomic_store(tmo, call, __ATOMIC_RELAXED, __HIP_MEMORY_SCOPE_AGENT); break; }
                __builtin_amdgcn_s_sleep(1);
            }
            __builtin_amdgcn_fence(__ATOMIC_ACQUIRE, "wavefront");
        }
        f32x4 o[16];
        asm volatile(
            "global_load_dwordx4 %0, %16, off sc0 sc1\n\tglobal_load_dwordx4 %1, %16, off offset:1024 sc0 sc1\n\t"
            "global_load_dwordx4 %2, %16, off offset:2048 sc0 sc1\n\tglobal_load_dwordx4 %3, %16, off offset:3072 sc0 sc1\n\t"
            "global_load_dwordx4 %4, %17, off sc0 sc1\n\tglobal_load_dwordx4 %5, %17, off offset:1024 sc0 sc1\n\t"
            "global_load_dwordx4 %6, %17, off offset:2048 sc0 sc1\n\tglobal_load_dwordx4 %7, %17, off offset:3072 sc0 sc1\n\t"
            "global_load_dwordx4 %8, %18, off sc0 sc1\n\tglobal_load_dwordx4 %9, %18, off offset:1024 sc0 sc1\n\t"
            "global_load_dwordx4 %10, %18, off offset:2048 sc0 sc1\n\tglobal_load_dwordx4 %11, %18, off offset:3072 sc0 sc1\n\t"
            "global_load_dwordx4 %12, %19, off sc0 sc1\n\tglobal_load_dwordx4 %13, %19, off offset:1024 sc0 sc1\n\t"
            "global_load_dwordx4 %14, %19, off offset:2048 sc0 sc1\n\tglobal_load_dwordx4 %15, %19, off offset:3072 sc0 sc1\n\t"
            "s_waitcnt vmcnt(0)"
            : "=&v"(o[0]), "=&v"(o[1]), "=&v"(o[2]), "=&v"(o[3]), "=&v"(o[4]), "=&v"(o[5]), "=&v"(o[6]), "=&v"(o[7]),
              "=&v"(o[8]), "=&v"(o[9]), "=&v"(o[10]), "=&v"(o[11]), "=&v"(o[12]), "=&v"(o[13]), "=&v"(o[14]), "=&v"(o[15])
            : "v"(theirs), "v"(theirs + 4096), "v"(theirs + 8192), "v"(theirs + 12288) : "memory");
#pragma unroll
        for (int f = 0; f < 16; ++f) acc[0][f >> 3][(f >> 1) & 3][f & 1] += o[f];
        const bool bad = (__hip_atomic_load(tmo, __ATOMIC_RELAXED, __HIP_MEMORY_SCOPE_AGENT) == call);
        if (d == rank) {
            store_dense(acc[0], obase + (size_t)(hh * HALF) * NTOT, bad);
        } else {
            const unsigned tot = pack_sparse(acc[0], wb, lane, bad);
            flush_sparse(wb, slot + hh * OBATCH, lane, tot);
            STAMP();
            flag_sparse(oflag + hh, lane, call, tot);
        }
    }
    STAMP();
    STAMP_END();
}

}

extern "C" void kernel_launch(int rank, int nranks, ncclComm_t comm,
                              hipStream_t stream,
                              void* const* d_in, const int* in_sizes, int n_in,
                              void* d_out, int out_size,
                              const int* in_dtypes, int out_dtype,
                              void* const* d_out_all,
                              void* const* d_ws_all, size_t ws_bytes) {
    if (nranks != NR || ws_bytes < WS_NEED) { fprintf(stderr, "unsupported configuration\n"); return; }
    static bool init = false;
    if (!init) {
        HIP_CHECK(hipFuncSetAttribute((const void*)fused_kernel, hipFuncAttributeMaxDynamicSharedMemorySize, LDS_BYTES));
        init = true;
    }
    Params p;
    p.x = (const float*)d_in[0];
    p.w = (const float*)d_in[1];
    for (int i = 0; i < NR; ++i) { p.out[i] = (float*)d_out_all[i]; p.ws[i] = (unsigned char*)d_ws_all[i]; }
    p.rank = rank;
    fused_kernel<<<256, 512, LDS_BYTES, stream>>>(p);
    HIP_CHECK(hipGetLastError());
}
```

```cpp
#include <hip/hip_runtime.h>
#include <rccl/rccl.h>
#include <cstdio>

#define HIP_CHECK(expr) do { hipError_t _e = (expr); if (_e != hipSuccess) { \
    fprintf(stderr, "HIP error %s at %s:%d\n", hipGetErrorString(_e), __FILE__, __LINE__); \
    return; } } while (0)

namespace {

#define LAS __attribute__((address_space(3)))
typedef _Float16 h8 __attribute__((ext_vector_type(8)));
typedef float f32x4 __attribute__((ext_vector_type(4)));
typedef unsigned u32x4 __attribute__((ext_vector_type(4)));

constexpr int NR = 8;
constexpr int MTOT = 4096;
constexpr int KS = 512;
constexpr int NTOT = 16384;
constexpr int NQ = NTOT / NR;
constexpr int BM = 256, BK = 64, HALF = 128;
constexpr int HTB = HALF * BK * 2;
constexpr int NMT = MTOT / BM;
constexpr int NNT = NQ / BM;
constexpr int NCH = KS / BK;
constexpr int NKT = NR * NCH;
constexpr int OBATCH = 512 + 16384;
constexpr int OSLOT = 2 * OBATCH;
constexpr int LDS_BYTES = 136 * 1024;
constexpr int CALLW = LDS_BYTES - 64;
static_assert(8 * (OBATCH + 256) <= CALLW && 8 * HTB <= CALLW, "lds");
#define OTHR 3
#define ROT 4
#define OTAIL 5
#define RSLEEP 2
#define NDEF 6

constexpr size_t OFF_CTR = 0;
constexpr size_t OFF_TMO = 256;
constexpr size_t OFF_Q = 1024;
constexpr int NPIECE = NCH * 24 * 8;
constexpr size_t OFF_FX = 4096;
constexpr size_t OFF_FW = OFF_FX + (size_t)NR * NCH * NMT * 128;
constexpr size_t OFF_DONE = OFF_FW + (size_t)NR * NCH * NNT * 128;
constexpr size_t OFF_STAGE = 1ull << 20;
constexpr size_t XA_BYTES = (size_t)NMT * NKT * 2 * HTB;
constexpr size_t WB_BYTES = (size_t)NNT * NKT * 2 * HTB;
constexpr size_t PAR_BYTES = XA_BYTES + WB_BYTES;
constexpr size_t OFF_OUT = OFF_STAGE + 2 * PAR_BYTES;
constexpr size_t OFF_PX = OFF_OUT + (size_t)NR * 128 * OSLOT + 4096;
constexpr size_t OFF_PXF = 384 * 1024;
constexpr size_t WS_NEED = OFF_PX + (size_t)64 * 2 * 8 * 16384;
static_assert(OFF_DONE + (size_t)NR * 128 * 64 <= OFF_PXF && OFF_PXF + 1024 * 64 <= 512 * 1024, "flag region");

#define STAMP() do {} while (0)
#define STAMP_INIT()
#define STAMP_BEGIN() do {} while (0)
#define STAMP_END() do {} while (0)
constexpr float WSCALE = 64.0f;
constexpr unsigned long long DEADLINE_TICKS = 40000000ull;

struct Params {
    const float* x;
    const float* w;
    float* out[NR];
    unsigned char* ws[NR];
    int rank;
};

__device__ __forceinline__ int lds_byte(int r, int c) {
    const int st = (r >> 4) * 2 + (c >> 5), rr = r & 15, cc = c & 31, ob = rr * 64 + cc * 2;
    return st * 1024 + (ob ^ (((ob >> 9) & 1) << 5));
}
__device__ __forceinline__ void stage_rc(int b, int& R, int& C) {
    const int st = b / 1024, sb = b % 1024, swz = sb ^ (((sb >> 9) & 1) << 5);
    R = (st >> 1) * 16 + swz / 64; C = (st & 1) * 32 + (swz % 64) / 2;
}
__device__ __forceinline__ unsigned ld_sys(const unsigned* p) {
    return __hip_atomic_load(p, __ATOMIC_RELAXED, __HIP_MEMORY_SCOPE_SYSTEM);
}
__device__ __forceinline__ h8 cvt8(const f32x4 a, const f32x4 b) {
    h8 r;
    r[0] = (_Float16)a[0]; r[1] = (_Float16)a[1]; r[2] = (_Float16)a[2]; r[3] = (_Float16)a[3];
    r[4] = (_Float16)b[0]; r[5] = (_Float16)b[1]; r[6] = (_Float16)b[2]; r[7] = (_Float16)b[3];
    return r;
}
template <class T> __device__ __forceinline__ void st_wt(void* ptr, const T v) {
    asm volatile("global_store_dwordx4 %0, %1, off sc0 sc1\n\ts_nop 1" :: "v"(ptr), "v"(v) : "memory");
}
#define WAIT_V(n) asm volatile("s_waitcnt vmcnt(" #n ")" ::: "memory")
#define WAIT_L(n) asm volatile("s_waitcnt lgkmcnt(" #n ")" ::: "memory")
#define BAR __builtin_amdgcn_s_barrier()
#define SCHED __builtin_amdgcn_sched_barrier(0)

__device__ __forceinline__ unsigned wave_incl_scan(unsigned x, int lane) {
#pragma unroll
    for (int o = 1; o < 64; o <<= 1) { const unsigned y = __shfl_up(x, o, 64); if (lane >= o) x += y; }
    return x;
}

__device__ __forceinline__ void wait_round(const unsigned* fxp, const unsigned* fwp, int j, unsigned call, unsigned* tmo, int lane) {
    const unsigned* fp = ((lane & 8) == 0 ? fxp + (size_t)(((lane & 7) * NCH + j) * NMT) * 32 : fwp + (size_t)(((lane & 7) * NCH + j) * NNT) * 32) + (lane >> 4) * 2;
    const unsigned long long t0 = __builtin_amdgcn_s_memrealtime();
    for (;;) {
        const unsigned long long v = __hip_atomic_load((const unsigned long long*)fp, __ATOMIC_RELAXED, __HIP_MEMORY_SCOPE_SYSTEM);
        if (__all((unsigned)v >= call && (unsigned)(v >> 32) >= call)) break;
        if (__hip_atomic_load(tmo, __ATOMIC_RELAXED, __HIP_MEMORY_SCOPE_AGENT) == call ||
            __builtin_amdgcn_s_memrealtime() - t0 > DEADLINE_TICKS) {
            __hip_atomic_store(tmo, call, __ATOMIC_RELAXED, __HIP_MEMORY_SCOPE_AGENT); break; }
        __builtin_amdgcn_s_sleep(RSLEEP);
    }
    __builtin_amdgcn_fence(__ATOMIC_ACQUIRE, "");
    asm volatile("s_waitcnt vmcnt(0)" ::: "memory");
}

__device__ __forceinline__ void store_dense(const f32x4 (&a)[2][4][2], float* obase, bool bad) {
    const float qn = __builtin_nanf("");
#pragma unroll
    for (int m = 0; m < 4; ++m)
#pragma unroll
        for (int bj = 0; bj < 2; ++bj)
#pragma unroll
            for (int n = 0; n < 2; ++n) {
                f32x4 v = a[bj][m][n];
#pragma unroll
                for (int e = 0; e < 4; ++e) v[e] = v[e] <= 0.f ? 0.f : v[e];
                if (bad) v = (f32x4){qn, qn, qn, qn};
                st_wt(obase + (size_t)(m * 16) * NTOT + bj * HALF + n * 16, v);
            }
}

__device__ __forceinline__ unsigned pack_sparse(const f32x4 (&a)[2][4][2], LAS unsigned char* wb, int lane, bool bad) {
    if (bad) {
        for (int i = lane; i < OBATCH / 4; i += 64) *(LAS unsigned*)(wb + i * 4) = 0xffffffffu;
        return 4096u;
    }
    const unsigned dummy = (unsigned)(OBATCH - 512) / 4u + (unsigned)lane;
    unsigned mw[2];
#pragma unroll
    for (int h = 0; h < 2; ++h) {
        unsigned mm = 0;
#pragma unroll
        for (int i = 31; i >= 0; --i) { const int f = h * 8 + (i >> 2); mm = mm + mm + ((a[f >> 3][(f >> 1) & 3][f & 1][i & 3] <= 0.f) ? 0u : 1u); }
        mw[h] = mm;
    }
    const unsigned cnt = __builtin_popcount(mw[0]) + __builtin_popcount(mw[1]);
    const unsigned inc = wave_incl_scan(cnt, lane);
    const unsigned tot = __builtin_amdgcn_readlane(inc, 63);
    *(LAS unsigned long long*)(wb + lane * 8) = ((unsigned long long)mw[1] << 32) | mw[0];
    unsigned pos = inc - cnt;
#pragma unroll
    for (int f = 0; f < 16; ++f) {
#pragma unroll
        for (int e = 0; e < 4; ++e) {
            const float x = a[f >> 3][(f >> 1) & 3][f & 1][e];
            const bool z = x <= 0.f;
            *(LAS float*)(wb + 512 + (z ? dummy : pos) * 4) = x;
            pos += z ? 0u : 1u;
        }
        if (f & 1) SCHED;
    }
    return tot;
}
__device__ __forceinline__ void flush_sparse(LAS unsigned char* wb, unsigned char* dst, int lane, unsigned tot) {
    const unsigned n16 = (512u + tot * 4u + 15u) >> 4;
    for (unsigned i = lane; i < n16; i += 64) {
        st_wt(dst + i * 16, *(LAS f32x4*)(wb + i * 16));
        if (i + OTAIL * 64 < n16) asm volatile("s_waitcnt vmcnt(%0)" :: "i"(OTHR) : "memory");
    }
}
__device__ __forceinline__ void flag_sparse(unsigned long long* flag, int lane, unsigned call, unsigned tot) {
    WAIT_V(0);
    if (lane == 0) __hip_atomic_store(flag, ((unsigned long long)tot << 32) | call, __ATOMIC_RELAXED, __HIP_MEMORY_SCOPE_SYSTEM);
}

__device__ __forceinline__ void recv_batch(unsigned char* myws, float* out, int src, int tl, int ai, int wid, int lane,
                                           LAS unsigned char* wl, unsigned call, unsigned* tmo) {
    const int wr = wid >> 2, wc = wid & 3, fr = lane & 15, fq = lane >> 4;
    const int sidx = (src * 16 + tl) * 8 + wid;
    const unsigned long long* fl = (const unsigned long long*)(myws + OFF_DONE + (size_t)sidx * 64) + ai;
    const unsigned char* sp = myws + OFF_OUT + (size_t)sidx * OSLOT + ai * OBATCH;
    float* obase = out + (size_t)((tl >> 3) * BM + ai * HALF + wr * 64 + fr) * NTOT + src * NQ + (tl & 7) * BM + wc * 32 + 4 * fq;
    const unsigned long long t0 = __builtin_amdgcn_s_memrealtime();
    bool bad = false;
    unsigned tot = 0;
    for (;;) {
        const unsigned long long v = __hip_atomic_load(fl, __ATOMIC_RELAXED, __HIP_MEMORY_SCOPE_SYSTEM);
        tot = __builtin_amdgcn_readfirstlane((unsigned)(v >> 32));
        if ((unsigned)__builtin_amdgcn_readfirstlane((unsigned)v) >= call) break;
        if (__hip_atomic_load(tmo, __ATOMIC_RELAXED, __HIP_MEMORY_SCOPE_AGENT) == call ||
            __builtin_amdgcn_s_memrealtime() - t0 > DEADLINE_TICKS) {
            __hip_atomic_store(tmo, call, __ATOMIC_RELAXED, __HIP_MEMORY_SCOPE_AGENT);
            bad = true; tot = 0; break;
        }
        __builtin_amdgcn_s_sleep(4);
    }
    tot = tot > 4096u ? 4096u : tot;
    __builtin_amdgcn_fence(__ATOMIC_ACQUIRE, "wavefront");
    const unsigned nit = (tot * 4u + 1023u) >> 10;
    for (unsigned i = 0; i < nit; ++i)
        __builtin_amdgcn_global_load_lds((const unsigned*)(sp + 512 + i * 1024 + lane * 16), (LAS unsigned*)(wl + i * 1024), 16, 0, 17);
    const unsigned long long mk = bad ? 0ull : __hip_atomic_load((const unsigned long long*)(sp + lane * 8), __ATOMIC_RELAXED, __HIP_MEMORY_SCOPE_SYSTEM);
    const unsigned mlo = (unsigned)mk, mhi = (unsigned)(mk >> 32);
    const unsigned cnt = __builtin_popcount(mlo) + __builtin_popcount(mhi);
    const unsigned inc = wave_incl_scan(cnt, lane);
    WAIT_V(0);
    unsigned pos = inc - cnt;
    f32x4 v[16];
#pragma unroll
    for (int f = 0; f < 16; ++f)
#pragma unroll
        for (int e = 0; e < 4; ++e) {
            const unsigned k = ((f < 8 ? mlo : mhi) >> ((f & 7) * 4 + e)) & 1u;
            const float x = *(LAS float*)(wl + pos * 4);
            v[f][e] = k ? x : 0.f;
            pos += k;
        }
    const float qn = __builtin_nanf("");
#pragma unroll
    for (int f = 0; f < 16; ++f) {
        if (bad) v[f] = (f32x4){qn, qn, qn, qn};
        *(f32x4*)(obase + (size_t)(((f >> 1) & 3) * 16) * NTOT + (f >> 3) * HALF + (f & 1) * 16) = v[f];
    }
}

__global__ void __launch_bounds__(512, 2) fused_kernel(Params p) {
    extern __shared__ __attribute__((aligned(16))) unsigned char smem[];
    LAS unsigned char* lds = (LAS unsigned char*)smem;
    const int tid = threadIdx.x;
    const int rank = p.rank;
    unsigned char* myws = p.ws[rank];
    unsigned* tmo = (unsigned*)(myws + OFF_TMO);

    STAMP_INIT();
    STAMP();
    const int xcd = blockIdx.x & 7, idx = blockIdx.x >> 3;
    const int d = (rank + 1 + ((xcd + ROT) & 7)) & 7;
    const int wid = __builtin_amdgcn_readfirstlane(tid >> 6), lane = tid & 63, wr = wid >> 2, wc = wid & 3, fr = lane & 15, fq = lane >> 4;
    unsigned call = 0;
    if (idx >= 8) {
        if (tid == 0) {
            unsigned tk = __hip_atomic_fetch_add((unsigned*)(myws + OFF_CTR), 1u, __ATOMIC_RELAXED, __HIP_MEMORY_SCOPE_AGENT);
            *(LAS unsigned*)(lds + CALLW) = tk / 192u + 1u;
        }
        __syncthreads();
        call = __builtin_amdgcn_readfirstlane(*(LAS unsigned*)(lds + CALLW));
        STAMP_BEGIN();
    }
    if (idx < 8) {
        unsigned char* dws = p.ws[d];
        LAS unsigned char* wl = lds + wid * OBATCH;
        unsigned* qctr = (unsigned*)(myws + OFF_Q + xcd * 128);
        auto piece_of = [&](int u, int& j, int& tile) {
            const int uu = u >> 3;
            constexpr int NN = NCH - NDEF;
            if (uu < NN * 24) { j = uu / 24; tile = uu % 24; tile = tile < 8 ? 16 + tile : tile - 8; }
            else if (uu < NN * 24 + NDEF * 16) { const int v = uu - NN * 24; j = NN + (v >> 4); const int sl = v & 15; tile = sl < 8 ? 16 + sl : (sl - 8) * 2; }
            else { const int v = uu - (NN * 24 + NDEF * 16); j = NN + (v >> 3); tile = (v & 7) * 2 + 1; }
        };
        f32x4 a[8];
        unsigned tk;
        const unsigned one = 1u;
        auto load_piece = [&](int u) {
            int j, tile; piece_of(u, j, tile);
            const int e = u & 7;
            const float* sp[8];
            if (tile < NMT) {
#pragma unroll
                for (int i = 0; i < 4; ++i) {
                    const int b = e * 4096 + i * 1024 + lane * 16;
                    int R, C; stage_rc(b & (HTB - 1), R, C);
                    sp[2 * i] = p.x + (size_t)(tile * BM + (b >> 14) * HALF + R) * KS + j * BK + C;
                    sp[2 * i + 1] = sp[2 * i] + 4;
                }
            } else {
                const float* src = p.w + (size_t)(j * BK + (lane >> 3) * 8) * NTOT + d * NQ + (tile - NMT) * BM + e * 32 + (lane & 7) * 4;
#pragma unroll
                for (int jj = 0; jj < 8; ++jj) sp[jj] = src + (size_t)jj * NTOT;
            }
            unsigned long long sv;
            asm volatile(
                "global_load_dwordx4 %0, %10, off\n\tglobal_load_dwordx4 %1, %11, off\n\tglobal_load_dwordx4 %2, %12, off\n\tglobal_load_dwordx4 %3, %13, off\n\t"
                "global_load_dwordx4 %4, %14, off\n\tglobal_load_dwordx4 %5, %15, off\n\tglobal_load_dwordx4 %6, %16, off\n\tglobal_load_dwordx4 %7, %17, off\n\t"
                "s_mov_b64 %9, exec\n\ts_mov_b64 exec, 1\n\tglobal_atomic_add %8, %18, %19, off sc0\n\ts_mov_b64 exec, %9"
                : "=&v"(a[0]), "=&v"(a[1]), "=&v"(a[2]), "=&v"(a[3]), "=&v"(a[4]), "=&v"(a[5]), "=&v"(a[6]), "=&v"(a[7]), "=&v"(tk), "=&s"(sv)
                : "v"(sp[0]), "v"(sp[1]), "v"(sp[2]), "v"(sp[3]), "v"(sp[4]), "v"(sp[5]), "v"(sp[6]), "v"(sp[7]), "v"(qctr), "v"(one) : "memory");
        };
#define LANDED(n) asm volatile("s_waitcnt vmcnt(" #n ")" : "+v"(a[0]), "+v"(a[1]), "+v"(a[2]), "+v"(a[3]), "+v"(a[4]), "+v"(a[5]), "+v"(a[6]), "+v"(a[7]), "+v"(tk) :: "memory")
        int u = idx * 8 + wid;
        load_piece(u);
        LANDED(0);
        call = (unsigned)__builtin_amdgcn_readfirstlane(tk) / (unsigned)NPIECE + 1u;
        const unsigned qbase = (call - 1u) * (unsigned)NPIECE;
        const size_t par_off = OFF_STAGE + (size_t)(call & 1u) * PAR_BYTES;
        while (u < NPIECE) {
            const unsigned nxt = (unsigned)__builtin_amdgcn_readfirstlane(tk) - qbase + 64u;
            int j, tile; piece_of(u, j, tile);
            const int e = u & 7, t = j * NR + rank;
            size_t foff;
            if (tile < NMT) {
                unsigned char* dst = dws + par_off + ((size_t)(tile * NKT + t) * 2) * HTB + e * 4096 + lane * 16;
#pragma unroll
                for (int i = 0; i < 4; ++i) st_wt(dst + i * 1024, cvt8(a[2 * i], a[2 * i + 1]));
                foff = OFF_FX + ((size_t)((rank * NCH + j) * NMT + tile)) * 128 + e * 4;
            } else {
                const int nt = tile - NMT, k0 = (lane >> 3) * 8, n0 = (lane & 7) * 4;
#pragma unroll
                for (int i = 0; i < 4; ++i) {
                    h8 r;
#pragma unroll
                    for (int jj = 0; jj < 8; ++jj) r[jj] = (_Float16)(a[jj][i] * WSCALE);
                    *(LAS h8*)(wl + lds_byte(n0 + i, k0)) = r;
                }
                unsigned char* dst = dws + par_off + XA_BYTES + ((size_t)(nt * NKT + t) * 2) * HTB + e * 4096 + lane * 16;
                h8 o[4];
#pragma unroll
                for (int i = 0; i < 4; ++i) o[i] = *(LAS h8*)(wl + i * 1024 + lane * 16);
#pragma unroll
                for (int i = 0; i < 4; ++i) st_wt(dst + i * 1024, o[i]);
                foff = OFF_FW + ((size_t)((rank * NCH + j) * NNT + nt)) * 128 + e * 4;
            }
            u = (int)nxt;
            if (u < NPIECE) {
                load_piece(u);
                WAIT_V(9);
            } else {
                if (lane == 0) __hip_atomic_fetch_add(qctr + 512, 1u, __ATOMIC_RELAXED, __HIP_MEMORY_SCOPE_AGENT);
                WAIT_V(0);
            }
            if (lane == 0) __hip_atomic_store((unsigned*)(dws + foff), call, __ATOMIC_RELAXED, __HIP_MEMORY_SCOPE_SYSTEM);
            LANDED(1);
            STAMP();
        }
        if (d == rank) return;
        for (int bt = 0; bt < 3; ++bt) {
            recv_batch(myws, p.out[rank], d, idx + (bt >> 1) * 8, bt & 1, wid, lane, wl, call, tmo);
            STAMP();
        }
        STAMP_END();
        return;
    }

    const size_t par_off = OFF_STAGE + (size_t)(call & 1u) * PAR_BYTES;
    const bool full = idx < 16;
    const int mt = 2 * d + (full ? 0 : 1);
    const int nt = idx & 7;
    const int hh = (idx >> 3) & 1;
    const unsigned voff = (unsigned)tid * 16u;
    const unsigned ldsw = (unsigned)wid * 1024u;
    const int aoff = lds_byte(wr * 64 + fr, fq * 8), boff = lds_byte(wc * 32 + fr, fq * 8);
    const unsigned char* cA = myws + par_off + (size_t)mt * NKT * 2 * HTB;
    const unsigned char* cB = myws + par_off + XA_BYTES + (size_t)nt * NKT * 2 * HTB;
    const unsigned* fxp = (const unsigned*)(myws + OFF_FX) + (size_t)mt * 32;
    const unsigned* fwp = (const unsigned*)(myws + OFF_FW) + (size_t)nt * 32;
    constexpr size_t kstep = 2 * HTB, hstep = HTB;
    const int sidx = (rank * 16 + (mt & 1) * 8 + nt) * 8 + wid;
    unsigned char* slot = p.ws[d] + OFF_OUT + (size_t)sidx * OSLOT;
    unsigned long long* oflag = (unsigned long long*)(p.ws[d] + OFF_DONE + (size_t)sidx * 64);
    float* obase = p.out[d] + (size_t)((mt & 1) * BM + wr * 64 + fr) * NTOT + rank * NQ + nt * BM + wc * 32 + 4 * fq;
    LAS unsigned char* wb = lds + wid * (OBATCH + 256);

#define STAGE(bufoff, gbase) do { _Pragma("unroll") for (int _i = 0; _i < 2; ++_i) \
        __builtin_amdgcn_global_load_lds((const unsigned*)((gbase) + voff + _i * 8192), (LAS unsigned*)(lds + (bufoff) + ldsw + _i * 8192), 16, 0, 0); } while (0)
#define LDFA(dst, off) do { _Pragma("unroll") for (int m = 0; m < 4; ++m) _Pragma("unroll") for (int k = 0; k < 2; ++k) dst[m][k] = *(const LAS h8*)(lds + (off) + aoff + m * 2048 + k * 1024); } while (0)
#define LDFB(dst, off) do { _Pragma("unroll") for (int n = 0; n < 2; ++n) _Pragma("unroll") for (int k = 0; k < 2; ++k) dst[n][k] = *(const LAS h8*)(lds + (off) + boff + n * 2048 + k * 1024); } while (0)
#define MMAX(C, At, Bt) do { _Pragma("unroll") for (int m = 0; m < 4; ++m) _Pragma("unroll") for (int n = 0; n < 2; ++n) _Pragma("unroll") for (int k = 0; k < 2; ++k) \
        C[m][n] = __builtin_amdgcn_mfma_f32_16x16x32_f16(Bt[n][k], At[m][k], C[m][n], 0, 0, 0); } while (0)
#define READY(j) do { if (tid < 64) wait_round(fxp, fwp, (j), call, tmo, lane); STAMP(); \
        asm volatile("" ::: "memory"); __builtin_amdgcn_s_barrier(); asm volatile("" ::: "memory"); } while (0)

#define SA(b, h) (((b) * 2 + (h)) * HTB)
#define SB(b, h) ((4 + (b) * 2 + (h)) * HTB)
#define MMA(ai, bj, At, Bt) do { __builtin_amdgcn_s_setprio(1); MMAX(acc[ai][bj], At, Bt); __builtin_amdgcn_s_setprio(0); } while (0)
    f32x4 acc[2][2][4][2];
#pragma unroll
    for (int a = 0; a < 2; ++a)
#pragma unroll
        for (int b = 0; b < 2; ++b)
#pragma unroll
            for (int m = 0; m < 4; ++m)
#pragma unroll
                for (int n = 0; n < 2; ++n) acc[a][b][m][n] = (f32x4){0.f, 0.f, 0.f, 0.f};
    h8 At[4][2], B0[2][2], B1[2][2];
    const int nti = full ? NKT : NKT / 2;
    const size_t tstep = full ? kstep : 2 * kstep;
    const int rsh = full ? 3 : 2;
    if (!full) { cA += (size_t)hh * kstep; cB += (size_t)hh * kstep; }
    READY(0);
    STAGE(SB(0, 0), cB); STAGE(SA(0, 0), cA); STAGE(SB(0, 1), cB + hstep); STAGE(SA(0, 1), cA + hstep);
    if (wr == 1) BAR;
    WAIT_V(4); BAR;
    STAGE(SB(1, 0), cB + tstep); STAGE(SA(1, 0), cA + tstep); STAGE(SB(1, 1), cB + hstep + tstep);
    WAIT_V(6); BAR;
    for (int t = 0; t < nti; t += 2) {
        const bool last = (t == nti - 2);
        const unsigned char* a1 = cA + (size_t)(t + 1) * tstep;
        const unsigned char* a2 = last ? cA : cA + (size_t)(t + 2) * tstep;
        const unsigned char* b2 = last ? cB : cB + (size_t)(t + 2) * tstep;
        const unsigned char* a3 = a2 + tstep; const unsigned char* b3 = b2 + tstep;
        if (!last && ((t + 2) & ((1 << rsh) - 1)) == 0) READY((t + 2) >> rsh);
        LDFB(B0, SB(0, 0)); SCHED; LDFA(At, SA(0, 0)); STAGE(SA(1, 1), a1 + hstep);
        WAIT_L(8); BAR; WAIT_L(0); MMA(0, 0, At, B0); BAR; SCHED;
        LDFB(B1, SB(0, 1)); STAGE(SB(0, 0), b2);
        BAR; WAIT_L(0); MMA(0, 1, At, B1); BAR;
        LDFA(At, SA(0, 1)); STAGE(SA(0, 0), a2);
        BAR; WAIT_L(0); MMA(1, 0, At, B0); BAR; SCHED;
        STAGE(SB(0, 1), b2 + hstep);
        WAIT_V(6); BAR; MMA(1, 1, At, B1); BAR;
        LDFB(B0, SB(1, 0)); SCHED; LDFA(At, SA(1, 0)); STAGE(SA(0, 1), a2 + hstep);
        WAIT_L(8); BAR; WAIT_L(0); MMA(0, 0, At, B0); BAR; SCHED;
        LDFB(B1, SB(1, 1)); STAGE(SB(1, 0), b3);
        BAR; WAIT_L(0); MMA(0, 1, At, B1); BAR;
        LDFA(At, SA(1, 1)); STAGE(SA(1, 0), a3);
        BAR; WAIT_L(0); MMA(1, 0, At, B0); BAR; SCHED;
        STAGE(SB(1, 1), b3 + hstep);
        WAIT_V(6); BAR; MMA(1, 1, At, B1); BAR;
    }
    WAIT_V(0);
    if (wr == 0) BAR;
    BAR;
    STAMP();
#pragma unroll
    for (int a = 0; a < 2; ++a)
#pragma unroll
        for (int b = 0; b < 2; ++b)
#pragma unroll
            for (int m = 0; m < 4; ++m)
#pragma unroll
                for (int n = 0; n < 2; ++n) acc[a][b][m][n] *= (1.0f / WSCALE);
    if (full) {
        const bool bad = (__hip_atomic_load(tmo, __ATOMIC_RELAXED, __HIP_MEMORY_SCOPE_AGENT) == call);
        if (d == rank) {
            store_dense(acc[0], obase, bad);
            store_dense(acc[1], obase + (size_t)HALF * NTOT, bad);
        } else {
            const unsigned tot0 = pack_sparse(acc[0], wb, lane, bad);
            if (wid == 0) {
                const unsigned* qctr = (const unsigned*)(myws + OFF_Q + xcd * 128) + 512;
                const unsigned long long t0 = __builtin_amdgcn_s_memrealtime();
                while ((unsigned)__builtin_amdgcn_readfirstlane(__hip_atomic_load(qctr, __ATOMIC_RELAXED, __HIP_MEMORY_SCOPE_AGENT)) < call * 64u &&
                       __builtin_amdgcn_s_memrealtime() - t0 < DEADLINE_TICKS)
                    __builtin_amdgcn_s_sleep(8);
            }
            asm volatile("" ::: "memory"); BAR; asm volatile("" ::: "memory");
            flush_sparse(wb, slot, lane, tot0);
            const unsigned tot1 = pack_sparse(acc[1], wb, lane, bad);
            flag_sparse(oflag, lane, call, tot0);
            flush_sparse(wb, slot + OBATCH, lane, tot1);
            STAMP();
            flag_sparse(oflag + 1, lane, call, tot1);
            STAMP();
            recv_batch(myws, p.out[rank], d, idx, 1, wid, lane, wb, call, tmo);
        }
    } else {
        const int px = (xcd * 8 + nt) * 2;
        unsigned char* mine = myws + OFF_PX + (size_t)((px + hh) * 8 + wid) * 16384 + lane * 16;
        const unsigned char* theirs = myws + OFF_PX + (size_t)((px + (hh ^ 1)) * 8 + wid) * 16384 + lane * 16;
        unsigned* pfl = (unsigned*)(myws + OFF_PXF);
#pragma unroll
        for (int f = 0; f < 16; ++f) {
            const f32x4 lo = acc[0][f >> 3][(f >> 1) & 3][f & 1], hi = acc[1][f >> 3][(f >> 1) & 3][f & 1];
            st_wt(mine + f * 1024, hh ? lo : hi);
            acc[0][f >> 3][(f >> 1) & 3][f & 1] = hh ? hi : lo;
        }
        WAIT_V(0);
        if (lane == 0) __hip_atomic_store(pfl + ((px + hh) * 8 + wid) * 16, call, __ATOMIC_RELAXED, __HIP_MEMORY_SCOPE_SYSTEM);
        {
            const unsigned* fl = pfl + ((px + (hh ^ 1)) * 8 + wid) * 16;
            const unsigned long long t0 = __builtin_amdgcn_s_memrealtime();
            while ((unsigned)__builtin_amdgcn_readfirstlane(ld_sys(fl)) < call) {
                if (__hip_atomic_load(tmo, __ATOMIC_RELAXED, __HIP_MEMORY_SCOPE_AGENT) == call ||
                    __builtin_amdgcn_s_memrealtime() - t0 > DEADLINE_TICKS) {
                    __hip_atomic_store(tmo, call, __ATOMIC_RELAXED, __HIP_MEMORY_SCOPE_AGENT); break; }
                __builtin_amdgcn_s_sleep(1);
            }
            __builtin_amdgcn_fence(__ATOMIC_ACQUIRE, "wavefront");
        }
        f32x4 o[16];
        asm volatile(
            "global_load_dwordx4 %0, %16, off sc0 sc1\n\tglobal_load_dwordx4 %1, %16, off offset:1024 sc0 sc1\n\t"
            "global_load_dwordx4 %2, %16, off offset:2048 sc0 sc1\n\tglobal_load_dwordx4 %3, %16, off offset:3072 sc0 sc1\n\t"
            "global_load_dwordx4 %4, %17, off sc0 sc1\n\tglobal_load_dwordx4 %5, %17, off offset:1024 sc0 sc1\n\t"
            "global_load_dwordx4 %6, %17, off offset:2048 sc0 sc1\n\tglobal_load_dwordx4 %7, %17, off offset:3072 sc0 sc1\n\t"
            "global_load_dwordx4 %8, %18, off sc0 sc1\n\tglobal_load_dwordx4 %9, %18, off offset:1024 sc0 sc1\n\t"
            "global_load_dwordx4 %10, %18, off offset:2048 sc0 sc1\n\tglobal_load_dwordx4 %11, %18, off offset:3072 sc0 sc1\n\t"
            "global_load_dwordx4 %12, %19, off sc0 sc1\n\tglobal_load_dwordx4 %13, %19, off offset:1024 sc0 sc1\n\t"
            "global_load_dwordx4 %14, %19, off offset:2048 sc0 sc1\n\tglobal_load_dwordx4 %15, %19, off offset:3072 sc0 sc1\n\t"
            "s_waitcnt vmcnt(0)"
            : "=&v"(o[0]), "=&v"(o[1]), "=&v"(o[2]), "=&v"(o[3]), "=&v"(o[4]), "=&v"(o[5]), "=&v"(o[6]), "=&v"(o[7]),
              "=&v"(o[8]), "=&v"(o[9]), "=&v"(o[10]), "=&v"(o[11]), "=&v"(o[12]), "=&v"(o[13]), "=&v"(o[14]), "=&v"(o[15])
            : "v"(theirs), "v"(theirs + 4096), "v"(theirs + 8192), "v"(theirs + 12288) : "memory");
#pragma unroll
        for (int f = 0; f < 16; ++f) acc[0][f >> 3][(f >> 1) & 3][f & 1] += o[f];
        const bool bad = (__hip_atomic_load(tmo, __ATOMIC_RELAXED, __HIP_MEMORY_SCOPE_AGENT) == call);
        if (d == rank) {
            store_dense(acc[0], obase + (size_t)(hh * HALF) * NTOT, bad);
        } else {
            const unsigned tot = pack_sparse(acc[0], wb, lane, bad);
            flush_sparse(wb, slot + hh * OBATCH, lane, tot);
            STAMP();
            flag_sparse(oflag + hh, lane, call, tot);
        }
    }
    STAMP();
    STAMP_END();
}

}

extern "C" void kernel_launch(int rank, int nranks, ncclComm_t comm,
                              hipStream_t stream,
                              void* const* d_in, const int* in_sizes, int n_in,
                              void* d_out, int out_size,
                              const int* in_dtypes, int out_dtype,
                              void* const* d_out_all,
                              void* const* d_ws_all, size_t ws_bytes) {
    if (nranks != NR || ws_bytes < WS_NEED) { fprintf(stderr, "unsupported configuration\n"); return; }
    static bool init = false;
    if (!init) {
        HIP_CHECK(hipFuncSetAttribute((const void*)fused_kernel, hipFuncAttributeMaxDynamicSharedMemorySize, LDS_BYTES));
        init = true;
    }
    Params p;
    p.x = (const float*)d_in[0];
    p.w = (const float*)d_in[1];
    for (int i = 0; i < NR; ++i) { p.out[i] = (float*)d_out_all[i]; p.ws[i] = (unsigned char*)d_ws_all[i]; }
    p.rank = rank;
    fused_kernel<<<256, 512, LDS_BYTES, stream>>>(p);
    HIP_CHECK(hipGetLastError());
}
```

```cpp
#include <hip/hip_runtime.h>
#include <rccl/rccl.h>
#include <cstdio>

#define HIP_CHECK(expr) do { hipError_t _e = (expr); if (_e != hipSuccess) { \
    fprintf(stderr, "HIP error %s at %s:%d\n", hipGetErrorString(_e), __FILE__, __LINE__); \
    return; } } while (0)

namespace {

#define LAS __attribute__((address_space(3)))
typedef _Float16 h8 __attribute__((ext_vector_type(8)));
typedef float f32x4 __attribute__((ext_vector_type(4)));
typedef unsigned u32x4 __attribute__((ext_vector_type(4)));

constexpr int NR = 8;
constexpr int MTOT = 4096;
constexpr int KS = 512;
constexpr int NTOT = 16384;
constexpr int NQ = NTOT / NR;
constexpr int BM = 256, BK = 64, HALF = 128;
constexpr int HTB = HALF * BK * 2;
constexpr int NMT = MTOT / BM;
constexpr int NNT = NQ / BM;
constexpr int NCH = KS / BK;
constexpr int NKT = NR * NCH;
constexpr int OBATCH = 512 + 16384;
constexpr int OSLOT = 2 * OBATCH;
constexpr int LDS_BYTES = 136 * 1024;
constexpr int CALLW = LDS_BYTES - 64;
static_assert(8 * (OBATCH + 256) <= CALLW && 8 * HTB <= CALLW, "lds");
#define OTHR 3
#define ROT 4
#define OTAIL 5
#define RSLEEP 2
#define NDEF 5

constexpr size_t OFF_CTR = 0;
constexpr size_t OFF_TMO = 256;
constexpr size_t OFF_Q = 1024;
constexpr int NPIECE = NCH * 24 * 8;
constexpr size_t OFF_FX = 4096;
constexpr size_t OFF_FW = OFF_FX + (size_t)NR * NCH * NMT * 128;
constexpr size_t OFF_DONE = OFF_FW + (size_t)NR * NCH * NNT * 128;
constexpr size_t OFF_STAGE = 1ull << 20;
constexpr size_t XA_BYTES = (size_t)NMT * NKT * 2 * HTB;
constexpr size_t WB_BYTES = (size_t)NNT * NKT * 2 * HTB;
constexpr size_t PAR_BYTES = XA_BYTES + WB_BYTES;
constexpr size_t OFF_OUT = OFF_STAGE + 2 * PAR_BYTES;
constexpr size_t OFF_PX = OFF_OUT + (size_t)NR * 128 * OSLOT + 4096;
constexpr size_t OFF_TC = 320 * 1024;
constexpr size_t OFF_PXF = 384 * 1024;
constexpr size_t WS_NEED = OFF_PX + (size_t)64 * 2 * 8 * 16384;
static_assert(OFF_DONE + (size_t)NR * 128 * 64 <= OFF_TC && OFF_TC + 8 * 192 * 4 <= OFF_PXF && OFF_PXF + 1024 * 64 <= 512 * 1024, "flag region");

#define STAMP() do {} while (0)
#define STAMP_INIT()
#define STAMP_BEGIN() do {} while (0)
#define STAMP_END() do {} while (0)
constexpr float WSCALE = 64.0f;
constexpr unsigned long long DEADLINE_TICKS = 40000000ull;

struct Params {
    const float* x;
    const float* w;
    float* out[NR];
    unsigned char* ws[NR];
    int rank;
};

__device__ __forceinline__ int lds_byte(int r, int c) {
    const int st = (r >> 4) * 2 + (c >> 5), rr = r & 15, cc = c & 31, ob = rr * 64 + cc * 2;
    return st * 1024 + (ob ^ (((ob >> 9) & 1) << 5));
}
__device__ __forceinline__ void stage_rc(int b, int& R, int& C) {
    const int st = b / 1024, sb = b % 1024, swz = sb ^ (((sb >> 9) & 1) << 5);
    R = (st >> 1) * 16 + swz / 64; C = (st & 1) * 32 + (swz % 64) / 2;
}
__device__ __forceinline__ unsigned ld_sys(const unsigned* p) {
    return __hip_atomic_load(p, __ATOMIC_RELAXED, __HIP_MEMORY_SCOPE_SYSTEM);
}
__device__ __forceinline__ h8 cvt8(const f32x4 a, const f32x4 b) {
    h8 r;
    r[0] = (_Float16)a[0]; r[1] = (_Float16)a[1]; r[2] = (_Float16)a[2]; r[3] = (_Float16)a[3];
    r[4] = (_Float16)b[0]; r[5] = (_Float16)b[1]; r[6] = (_Float16)b[2]; r[7] = (_Float16)b[3];
    return r;
}
template <class T> __device__ __forceinline__ void st_wt(void* ptr, const T v) {
    asm volatile("global_store_dwordx4 %0, %1, off sc0 sc1\n\ts_nop 1" :: "v"(ptr), "v"(v) : "memory");
}
#define WAIT_V(n) asm volatile("s_waitcnt vmcnt(" #n ")" ::: "memory")
#define WAIT_L(n) asm volatile("s_waitcnt lgkmcnt(" #n ")" ::: "memory")
#define BAR __builtin_amdgcn_s_barrier()
#define SCHED __builtin_amdgcn_sched_barrier(0)

__device__ __forceinline__ unsigned wave_incl_scan(unsigned x, int lane) {
#pragma unroll
    for (int o = 1; o < 64; o <<= 1) { const unsigned y = __shfl_up(x, o, 64); if (lane >= o) x += y; }
    return x;
}

__device__ __forceinline__ void wait_round(const unsigned* fxp, const unsigned* fwp, int j, unsigned call, unsigned* tmo, int lane) {
    const unsigned* fp = (lane & 8) == 0 ? fxp + (size_t)(((lane & 7) * NCH + j) * NMT) * 32 : fwp + (size_t)(((lane & 7) * NCH + j) * NNT) * 32;
    const unsigned long long t0 = __builtin_amdgcn_s_memrealtime();
    for (;;) {
        const unsigned v = lane < 16 ? ld_sys(fp) : call;
        if (__all(v >= call)) break;
        if (__hip_atomic_load(tmo, __ATOMIC_RELAXED, __HIP_MEMORY_SCOPE_AGENT) == call ||
            __builtin_amdgcn_s_memrealtime() - t0 > DEADLINE_TICKS) {
            __hip_atomic_store(tmo, call, __ATOMIC_RELAXED, __HIP_MEMORY_SCOPE_AGENT); break; }
        __builtin_amdgcn_s_sleep(RSLEEP);
    }
    __builtin_amdgcn_fence(__ATOMIC_ACQUIRE, "");
    asm volatile("s_waitcnt vmcnt(0)" ::: "memory");
}

__device__ __forceinline__ void store_dense(const f32x4 (&a)[2][4][2], float* obase, bool bad) {
    const float qn = __builtin_nanf("");
#pragma unroll
    for (int m = 0; m < 4; ++m)
#pragma unroll
        for (int bj = 0; bj < 2; ++bj)
#pragma unroll
            for (int n = 0; n < 2; ++n) {
                f32x4 v = a[bj][m][n];
#pragma unroll
                for (int e = 0; e < 4; ++e) v[e] = v[e] <= 0.f ? 0.f : v[e];
                if (bad) v = (f32x4){qn, qn, qn, qn};
                st_wt(obase + (size_t)(m * 16) * NTOT + bj * HALF + n * 16, v);
            }
}

__device__ __forceinline__ unsigned pack_sparse(const f32x4 (&a)[2][4][2], LAS unsigned char* wb, int lane, bool bad) {
    if (bad) {
        for (int i = lane; i < OBATCH / 4; i += 64) *(LAS unsigned*)(wb + i * 4) = 0xffffffffu;
        return 4096u;
    }
    const unsigned dummy = (unsigned)(OBATCH - 512) / 4u + (unsigned)lane;
    unsigned mw[2];
#pragma unroll
    for (int h = 0; h < 2; ++h) {
        unsigned mm = 0;
#pragma unroll
        for (int i = 31; i >= 0; --i) { const int f = h * 8 + (i >> 2); mm = mm + mm + ((a[f >> 3][(f >> 1) & 3][f & 1][i & 3] <= 0.f) ? 0u : 1u); }
        mw[h] = mm;
    }
    const unsigned cnt = __builtin_popcount(mw[0]) + __builtin_popcount(mw[1]);
    const unsigned inc = wave_incl_scan(cnt, lane);
    const unsigned tot = __builtin_amdgcn_readlane(inc, 63);
    *(LAS unsigned long long*)(wb + lane * 8) = ((unsigned long long)mw[1] << 32) | mw[0];
    unsigned pos = inc - cnt;
#pragma unroll
    for (int f = 0; f < 16; ++f) {
#pragma unroll
        for (int e = 0; e < 4; ++e) {
            const float x = a[f >> 3][(f >> 1) & 3][f & 1][e];
            const bool z = x <= 0.f;
            *(LAS float*)(wb + 512 + (z ? dummy : pos) * 4) = x;
            pos += z ? 0u : 1u;
        }
        if (f & 1) SCHED;
    }
    return tot;
}
__device__ __forceinline__ void flush_sparse(LAS unsigned char* wb, unsigned char* dst, int lane, unsigned tot) {
    const unsigned n16 = (512u + tot * 4u + 15u) >> 4;
    for (unsigned i = lane; i < n16; i += 64) {
        st_wt(dst + i * 16, *(LAS f32x4*)(wb + i * 16));
        if (i + OTAIL * 64 < n16) asm volatile("s_waitcnt vmcnt(%0)" :: "i"(OTHR) : "memory");
    }
}
__device__ __forceinline__ void flag_sparse(unsigned long long* flag, int lane, unsigned call, unsigned tot) {
    WAIT_V(0);
    if (lane == 0) __hip_atomic_store(flag, ((unsigned long long)tot << 32) | call, __ATOMIC_RELAXED, __HIP_MEMORY_SCOPE_SYSTEM);
}

__device__ __forceinline__ void recv_batch(unsigned char* myws, float* out, int src, int tl, int ai, int wid, int lane,
                                           LAS unsigned char* wl, unsigned call, unsigned* tmo) {
    const int wr = wid >> 2, wc = wid & 3, fr = lane & 15, fq = lane >> 4;
    const int sidx = (src * 16 + tl) * 8 + wid;
    const unsigned long long* fl = (const unsigned long long*)(myws + OFF_DONE + (size_t)sidx * 64) + ai;
    const unsigned char* sp = myws + OFF_OUT + (size_t)sidx * OSLOT + ai * OBATCH;
    float* obase = out + (size_t)((tl >> 3) * BM + ai * HALF + wr * 64 + fr) * NTOT + src * NQ + (tl & 7) * BM + wc * 32 + 4 * fq;
    const unsigned long long t0 = __builtin_amdgcn_s_memrealtime();
    bool bad = false;
    unsigned tot = 0;
    for (;;) {
        const unsigned long long v = __hip_atomic_load(fl, __ATOMIC_RELAXED, __HIP_MEMORY_SCOPE_SYSTEM);
        tot = __builtin_amdgcn_readfirstlane((unsigned)(v >> 32));
        if ((unsigned)__builtin_amdgcn_readfirstlane((unsigned)v) >= call) break;
        if (__hip_atomic_load(tmo, __ATOMIC_RELAXED, __HIP_MEMORY_SCOPE_AGENT) == call ||
            __builtin_amdgcn_s_memrealtime() - t0 > DEADLINE_TICKS) {
            __hip_atomic_store(tmo, call, __ATOMIC_RELAXED, __HIP_MEMORY_SCOPE_AGENT);
            bad = true; tot = 0; break;
        }
        __builtin_amdgcn_s_sleep(4);
    }
    tot = tot > 4096u ? 4096u : tot;
    __builtin_amdgcn_fence(__ATOMIC_ACQUIRE, "wavefront");
    const unsigned nit = (tot * 4u + 1023u) >> 10;
    for (unsigned i = 0; i < nit; ++i)
        __builtin_amdgcn_global_load_lds((const unsigned*)(sp + 512 + i * 1024 + lane * 16), (LAS unsigned*)(wl + i * 1024), 16, 0, 17);
    const unsigned long long mk = bad ? 0ull : __hip_atomic_load((const unsigned long long*)(sp + lane * 8), __ATOMIC_RELAXED, __HIP_MEMORY_SCOPE_SYSTEM);
    const unsigned mlo = (unsigned)mk, mhi = (unsigned)(mk >> 32);
    const unsigned cnt = __builtin_popcount(mlo) + __builtin_popcount(mhi);
    const unsigned inc = wave_incl_scan(cnt, lane);
    WAIT_V(0);
    unsigned pos = inc - cnt;
    f32x4 v[16];
#pragma unroll
    for (int f = 0; f < 16; ++f)
#pragma unroll
        for (int e = 0; e < 4; ++e) {
            const unsigned k = ((f < 8 ? mlo : mhi) >> ((f & 7) * 4 + e)) & 1u;
            const float x = *(LAS float*)(wl + pos * 4);
            v[f][e] = k ? x : 0.f;
            pos += k;
        }
    const float qn = __builtin_nanf("");
#pragma unroll
    for (int f = 0; f < 16; ++f) {
        if (bad) v[f] = (f32x4){qn, qn, qn, qn};
        *(f32x4*)(obase + (size_t)(((f >> 1) & 3) * 16) * NTOT + (f >> 3) * HALF + (f & 1) * 16) = v[f];
    }
}

__global__ void __launch_bounds__(512, 2) fused_kernel(Params p) {
    extern __shared__ __attribute__((aligned(16))) unsigned char smem[];
    LAS unsigned char* lds = (LAS unsigned char*)smem;
    const int tid = threadIdx.x;
    const int rank = p.rank;
    unsigned char* myws = p.ws[rank];
    unsigned* tmo = (unsigned*)(myws + OFF_TMO);

    STAMP_INIT();
    STAMP();
    const int xcd = blockIdx.x & 7, idx = blockIdx.x >> 3;
    const int d = (rank + 1 + ((xcd + ROT) & 7)) & 7;
    const int wid = __builtin_amdgcn_readfirstlane(tid >> 6), lane = tid & 63, wr = wid >> 2, wc = wid & 3, fr = lane & 15, fq = lane >> 4;
    unsigned call = 0;
    if (idx >= 8) {
        if (tid == 0) {
            unsigned tk = __hip_atomic_fetch_add((unsigned*)(myws + OFF_CTR), 1u, __ATOMIC_RELAXED, __HIP_MEMORY_SCOPE_AGENT);
            *(LAS unsigned*)(lds + CALLW) = tk / 192u + 1u;
        }
        __syncthreads();
        call = __builtin_amdgcn_readfirstlane(*(LAS unsigned*)(lds + CALLW));
        STAMP_BEGIN();
    }
    if (idx < 8) {
        unsigned char* dws = p.ws[d];
        LAS unsigned char* wl = lds + wid * OBATCH;
        unsigned* qctr = (unsigned*)(myws + OFF_Q + xcd * 128);
        auto piece_of = [&](int u, int& j, int& tile) {
            const int uu = u >> 3;
            constexpr int NN = NCH - NDEF;
            if (uu < NN * 24) { j = uu / 24; tile = uu % 24; tile = tile < 8 ? 16 + tile : tile - 8; }
            else if (uu < NN * 24 + NDEF * 16) { const int v = uu - NN * 24; j = NN + (v >> 4); const int sl = v & 15; tile = sl < 8 ? 16 + sl : (sl - 8) * 2; }
            else { const int v = uu - (NN * 24 + NDEF * 16); j = NN + (v >> 3); tile = (v & 7) * 2 + 1; }
        };
        f32x4 a[8];
        unsigned tk;
        const unsigned one = 1u;
        auto load_piece = [&](int u) {
            int j, tile; piece_of(u, j, tile);
            const int e = u & 7;
            const float* sp[8];
            if (tile < NMT) {
#pragma unroll
                for (int i = 0; i < 4; ++i) {
                    const int b = e * 4096 + i * 1024 + lane * 16;
                    int R, C; stage_rc(b & (HTB - 1), R, C);
                    sp[2 * i] = p.x + (size_t)(tile * BM + (b >> 14) * HALF + R) * KS + j * BK + C;
                    sp[2 * i + 1] = sp[2 * i] + 4;
                }
            } else {
                const float* src = p.w + (size_t)(j * BK + (lane >> 3) * 8) * NTOT + d * NQ + (tile - NMT) * BM + e * 32 + (lane & 7) * 4;
#pragma unroll
                for (int jj = 0; jj < 8; ++jj) sp[jj] = src + (size_t)jj * NTOT;
            }
            unsigned long long sv;
            asm volatile(
                "global_load_dwordx4 %0, %10, off\n\tglobal_load_dwordx4 %1, %11, off\n\tglobal_load_dwordx4 %2, %12, off\n\tglobal_load_dwordx4 %3, %13, off\n\t"
                "global_load_dwordx4 %4, %14, off\n\tglobal_load_dwordx4 %5, %15, off\n\tglobal_load_dwordx4 %6, %16, off\n\tglobal_load_dwordx4 %7, %17, off\n\t"
                "s_mov_b64 %9, exec\n\ts_mov_b64 exec, 1\n\tglobal_atomic_add %8, %18, %19, off sc0\n\ts_mov_b64 exec, %9"
                : "=&v"(a[0]), "=&v"(a[1]), "=&v"(a[2]), "=&v"(a[3]), "=&v"(a[4]), "=&v"(a[5]), "=&v"(a[6]), "=&v"(a[7]), "=&v"(tk), "=&s"(sv)
                : "v"(sp[0]), "v"(sp[1]), "v"(sp[2]), "v"(sp[3]), "v"(sp[4]), "v"(sp[5]), "v"(sp[6]), "v"(sp[7]), "v"(qctr), "v"(one) : "memory");
        };
#define LANDED(n) asm volatile("s_waitcnt vmcnt(" #n ")" : "+v"(a[0]), "+v"(a[1]), "+v"(a[2]), "+v"(a[3]), "+v"(a[4]), "+v"(a[5]), "+v"(a[6]), "+v"(a[7]), "+v"(tk) :: "memory")
        int u = idx * 8 + wid;
        load_piece(u);
        LANDED(0);
        call = (unsigned)__builtin_amdgcn_readfirstlane(tk) / (unsigned)NPIECE + 1u;
        const unsigned qbase = (call - 1u) * (unsigned)NPIECE;
        const size_t par_off = OFF_STAGE + (size_t)(call & 1u) * PAR_BYTES;
        unsigned cn = 0; size_t pfoff = 0; bool pend = false;
        unsigned* tcnt = (unsigned*)(myws + OFF_TC) + xcd * (NPIECE / 8);
#define SEND_FLAG(n) do { if (pend) { asm volatile("s_waitcnt vmcnt(" #n ")" : "+v"(cn) :: "memory"); \
            if (((unsigned)__builtin_amdgcn_readfirstlane(cn) & 7u) == 7u && lane == 0) \
                __hip_atomic_store((unsigned*)(dws + pfoff), call, __ATOMIC_RELAXED, __HIP_MEMORY_SCOPE_SYSTEM); } } while (0)
        while (u < NPIECE) {
            const unsigned nxt = (unsigned)__builtin_amdgcn_readfirstlane(tk) - qbase + 64u;
            unsigned* tc = tcnt + (u >> 3);
            int j, tile; piece_of(u, j, tile);
            const int e = u & 7, t = j * NR + rank;
            size_t foff;
            if (tile < NMT) {
                unsigned char* dst = dws + par_off + ((size_t)(tile * NKT + t) * 2) * HTB + e * 4096 + lane * 16;
#pragma unroll
                for (int i = 0; i < 4; ++i) st_wt(dst + i * 1024, cvt8(a[2 * i], a[2 * i + 1]));
                foff = OFF_FX + ((size_t)((rank * NCH + j) * NMT + tile)) * 128;
            } else {
                const int nt = tile - NMT, k0 = (lane >> 3) * 8, n0 = (lane & 7) * 4;
#pragma unroll
                for (int i = 0; i < 4; ++i) {
                    h8 r;
#pragma unroll
                    for (int jj = 0; jj < 8; ++jj) r[jj] = (_Float16)(a[jj][i] * WSCALE);
                    *(LAS h8*)(wl + lds_byte(n0 + i, k0)) = r;
                }
                unsigned char* dst = dws + par_off + XA_BYTES + ((size_t)(nt * NKT + t) * 2) * HTB + e * 4096 + lane * 16;
                h8 o[4];
#pragma unroll
                for (int i = 0; i < 4; ++i) o[i] = *(LAS h8*)(wl + i * 1024 + lane * 16);
#pragma unroll
                for (int i = 0; i < 4; ++i) st_wt(dst + i * 1024, o[i]);
                foff = OFF_FW + ((size_t)((rank * NCH + j) * NNT + nt)) * 128;
            }
            SEND_FLAG(4);
            u = (int)nxt;
            if (u < NPIECE) {
                load_piece(u);
                WAIT_V(9);
            } else {
                if (lane == 0) __hip_atomic_fetch_add(qctr + 512, 1u, __ATOMIC_RELAXED, __HIP_MEMORY_SCOPE_AGENT);
                WAIT_V(0);
            }
            {
                unsigned long long sv;
                asm volatile("s_mov_b64 %1, exec\n\ts_mov_b64 exec, 1\n\tglobal_atomic_add %0, %2, %3, off sc0\n\ts_mov_b64 exec, %1"
                             : "=&v"(cn), "=&s"(sv) : "v"(tc), "v"(one) : "memory");
                pfoff = foff; pend = true;
            }
            LANDED(1);
            STAMP();
        }
        SEND_FLAG(0);
        if (d == rank) return;
        for (int bt = 0; bt < 3; ++bt) {
            recv_batch(myws, p.out[rank], d, idx + (bt >> 1) * 8, bt & 1, wid, lane, wl, call, tmo);
            STAMP();
        }
        STAMP_END();
        return;
    }

    const size_t par_off = OFF_STAGE + (size_t)(call & 1u) * PAR_BYTES;
    const bool full = idx < 16;
    const int mt = 2 * d + (full ? 0 : 1);
    const int nt = idx & 7;
    const int hh = (idx >> 3) & 1;
    const unsigned voff = (unsigned)tid * 16u;
    const unsigned ldsw = (unsigned)wid * 1024u;
    const int aoff = lds_byte(wr * 64 + fr, fq * 8), boff = lds_byte(wc * 32 + fr, fq * 8);
    const unsigned char* cA = myws + par_off + (size_t)mt * NKT * 2 * HTB;
    const unsigned char* cB = myws + par_off + XA_BYTES + (size_t)nt * NKT * 2 * HTB;
    const unsigned* fxp = (const unsigned*)(myws + OFF_FX) + (size_t)mt * 32;
    const unsigned* fwp = (const unsigned*)(myws + OFF_FW) + (size_t)nt * 32;
    constexpr size_t kstep = 2 * HTB, hstep = HTB;
    const int sidx = (rank * 16 + (mt & 1) * 8 + nt) * 8 + wid;
    unsigned char* slot = p.ws[d] + OFF_OUT + (size_t)sidx * OSLOT;
    unsigned long long* oflag = (unsigned long long*)(p.ws[d] + OFF_DONE + (size_t)sidx * 64);
    float* obase = p.out[d] + (size_t)((mt & 1) * BM + wr * 64 + fr) * NTOT + rank * NQ + nt * BM + wc * 32 + 4 * fq;
    LAS unsigned char* wb = lds + wid * (OBATCH + 256);

#define STAGE(bufoff, gbase) do { _Pragma("unroll") for (int _i = 0; _i < 2; ++_i) \
        __builtin_amdgcn_global_load_lds((const unsigned*)((gbase) + voff + _i * 8192), (LAS unsigned*)(lds + (bufoff) + ldsw + _i * 8192), 16, 0, 0); } while (0)
#define LDFA(dst, off) do { _Pragma("unroll") for (int m = 0; m < 4; ++m) _Pragma("unroll") for (int k = 0; k < 2; ++k) dst[m][k] = *(const LAS h8*)(lds + (off) + aoff + m * 2048 + k * 1024); } while (0)
#define LDFB(dst, off) do { _Pragma("unroll") for (int n = 0; n < 2; ++n) _Pragma("unroll") for (int k = 0; k < 2; ++k) dst[n][k] = *(const LAS h8*)(lds + (off) + boff + n * 2048 + k * 1024); } while (0)
#define MMAX(C, At, Bt) do { _Pragma("unroll") for (int m = 0; m < 4; ++m) _Pragma("unroll") for (int n = 0; n < 2; ++n) _Pragma("unroll") for (int k = 0; k < 2; ++k) \
        C[m][n] = __builtin_amdgcn_mfma_f32_16x16x32_f16(Bt[n][k], At[m][k], C[m][n], 0, 0, 0); } while (0)
#define READY(j) do { if (tid < 64) wait_round(fxp, fwp, (j), call, tmo, lane); STAMP(); \
        asm volatile("" ::: "memory"); __builtin_amdgcn_s_barrier(); asm volatile("" ::: "memory"); } while (0)

#define SA(b, h) (((b) * 2 + (h)) * HTB)
#define SB(b, h) ((4 + (b) * 2 + (h)) * HTB)
#define MMA(ai, bj, At, Bt) do { __builtin_amdgcn_s_setprio(1); MMAX(acc[ai][bj], At, Bt); __builtin_amdgcn_s_setprio(0); } while (0)
    f32x4 acc[2][2][4][2];
#pragma unroll
    for (int a = 0; a < 2; ++a)
#pragma unroll
        for (int b = 0; b < 2; ++b)
#pragma unroll
            for (int m = 0; m < 4; ++m)
#pragma unroll
                for (int n = 0; n < 2; ++n) acc[a][b][m][n] = (f32x4){0.f, 0.f, 0.f, 0.f};
    h8 At[4][2], B0[2][2], B1[2][2];
    const int nti = full ? NKT : NKT / 2;
    const size_t tstep = full ? kstep : 2 * kstep;
    const int rsh = full ? 3 : 2;
    if (!full) { cA += (size_t)hh * kstep; cB += (size_t)hh * kstep; }
    READY(0);
    STAGE(SB(0, 0), cB); STAGE(SA(0, 0), cA); STAGE(SB(0, 1), cB + hstep); STAGE(SA(0, 1), cA + hstep);
    if (wr == 1) BAR;
    WAIT_V(4); BAR;
    STAGE(SB(1, 0), cB + tstep); STAGE(SA(1, 0), cA + tstep); STAGE(SB(1, 1), cB + hstep + tstep);
    WAIT_V(6); BAR;
    for (int t = 0; t < nti; t += 2) {
        const bool last = (t == nti - 2);
        const unsigned char* a1 = cA + (size_t)(t + 1) * tstep;
        const unsigned char* a2 = last ? cA : cA + (size_t)(t + 2) * tstep;
        const unsigned char* b2 = last ? cB : cB + (size_t)(t + 2) * tstep;
        const unsigned char* a3 = a2 + tstep; const unsigned char* b3 = b2 + tstep;
        if (!last && ((t + 2) & ((1 << rsh) - 1)) == 0) READY((t + 2) >> rsh);
        LDFB(B0, SB(0, 0)); SCHED; LDFA(At, SA(0, 0)); STAGE(SA(1, 1), a1 + hstep);
        WAIT_L(8); BAR; WAIT_L(0); MMA(0, 0, At, B0); BAR; SCHED;
        LDFB(B1, SB(0, 1)); STAGE(SB(0, 0), b2);
        BAR; WAIT_L(0); MMA(0, 1, At, B1); BAR;
        LDFA(At, SA(0, 1)); STAGE(SA(0, 0), a2);
        BAR; WAIT_L(0); MMA(1, 0, At, B0); BAR; SCHED;
        STAGE(SB(0, 1), b2 + hstep);
        WAIT_V(6); BAR; MMA(1, 1, At, B1); BAR;
        LDFB(B0, SB(1, 0)); SCHED; LDFA(At, SA(1, 0)); STAGE(SA(0, 1), a2 + hstep);
        WAIT_L(8); BAR; WAIT_L(0); MMA(0, 0, At, B0); BAR; SCHED;
        LDFB(B1, SB(1, 1)); STAGE(SB(1, 0), b3);
        BAR; WAIT_L(0); MMA(0, 1, At, B1); BAR;
        LDFA(At, SA(1, 1)); STAGE(SA(1, 0), a3);
        BAR; WAIT_L(0); MMA(1, 0, At, B0); BAR; SCHED;
        STAGE(SB(1, 1), b3 + hstep);
        WAIT_V(6); BAR; MMA(1, 1, At, B1); BAR;
    }
    WAIT_V(0);
    if (wr == 0) BAR;
    BAR;
    STAMP();
#pragma unroll
    for (int a = 0; a < 2; ++a)
#pragma unroll
        for (int b = 0; b < 2; ++b)
#pragma unroll
            for (int m = 0; m < 4; ++m)
#pragma unroll
                for (int n = 0; n < 2; ++n) acc[a][b][m][n] *= (1.0f / WSCALE);
    if (full) {
        const bool bad = (__hip_atomic_load(tmo, __ATOMIC_RELAXED, __HIP_MEMORY_SCOPE_AGENT) == call);
        if (d == rank) {
            store_dense(acc[0], obase, bad);
            store_dense(acc[1], obase + (size_t)HALF * NTOT, bad);
        } else {
            const unsigned tot0 = pack_sparse(acc[0], wb, lane, bad);
            if (wid == 0) {
                const unsigned* qctr = (const unsigned*)(myws + OFF_Q + xcd * 128) + 512;
                const unsigned long long t0 = __builtin_amdgcn_s_memrealtime();
                while ((unsigned)__builtin_amdgcn_readfirstlane(__hip_atomic_load(qctr, __ATOMIC_RELAXED, __HIP_MEMORY_SCOPE_AGENT)) < call * 64u &&
                       __builtin_amdgcn_s_memrealtime() - t0 < DEADLINE_TICKS)
                    __builtin_amdgcn_s_sleep(8);
            }
            asm volatile("" ::: "memory"); BAR; asm volatile("" ::: "memory");
            flush_sparse(wb, slot, lane, tot0);
            const unsigned tot1 = pack_sparse(acc[1], wb, lane, bad);
            flag_sparse(oflag, lane, call, tot0);
            flush_sparse(wb, slot + OBATCH, lane, tot1);
            STAMP();
            flag_sparse(oflag + 1, lane, call, tot1);
            STAMP();
            recv_batch(myws, p.out[rank], d, idx, 1, wid, lane, wb, call, tmo);
        }
    } else {
        const int px = (xcd * 8 + nt) * 2;
        unsigned char* mine = myws + OFF_PX + (size_t)((px + hh) * 8 + wid) * 16384 + lane * 16;
        const unsigned char* theirs = myws + OFF_PX + (size_t)((px + (hh ^ 1)) * 8 + wid) * 16384 + lane * 16;
        unsigned* pfl = (unsigned*)(myws + OFF_PXF);
#pragma unroll
        for (int f = 0; f < 16; ++f) {
            const f32x4 lo = acc[0][f >> 3][(f >> 1) & 3][f & 1], hi = acc[1][f >> 3][(f >> 1) & 3][f & 1];
            st_wt(mine + f * 1024, hh ? lo : hi);
            acc[0][f >> 3][(f >> 1) & 3][f & 1] = hh ? hi : lo;
        }
        WAIT_V(0);
        if (lane == 0) __hip_atomic_store(pfl + ((px + hh) * 8 + wid) * 16, call, __ATOMIC_RELAXED, __HIP_MEMORY_SCOPE_SYSTEM);
        {
            const unsigned* fl = pfl + ((px + (hh ^ 1)) * 8 + wid) * 16;
            const unsigned long long t0 = __builtin_amdgcn_s_memrealtime();
            while ((unsigned)__builtin_amdgcn_readfirstlane(ld_sys(fl)) < call) {
                if (__hip_atomic_load(tmo, __ATOMIC_RELAXED, __HIP_MEMORY_SCOPE_AGENT) == call ||
                    __builtin_amdgcn_s_memrealtime() - t0 > DEADLINE_TICKS) {
                    __hip_atomic_store(tmo, call, __ATOMIC_RELAXED, __HIP_MEMORY_SCOPE_AGENT); break; }
                __builtin_amdgcn_s_sleep(1);
            }
            __builtin_amdgcn_fence(__ATOMIC_ACQUIRE, "wavefront");
        }
        f32x4 o[16];
        asm volatile(
            "global_load_dwordx4 %0, %16, off sc0 sc1\n\tglobal_load_dwordx4 %1, %16, off offset:1024 sc0 sc1\n\t"
            "global_load_dwordx4 %2, %16, off offset:2048 sc0 sc1\n\tglobal_load_dwordx4 %3, %16, off offset:3072 sc0 sc1\n\t"
            "global_load_dwordx4 %4, %17, off sc0 sc1\n\tglobal_load_dwordx4 %5, %17, off offset:1024 sc0 sc1\n\t"
            "global_load_dwordx4 %6, %17, off offset:2048 sc0 sc1\n\tglobal_load_dwordx4 %7, %17, off offset:3072 sc0 sc1\n\t"
            "global_load_dwordx4 %8, %18, off sc0 sc1\n\tglobal_load_dwordx4 %9, %18, off offset:1024 sc0 sc1\n\t"
            "global_load_dwordx4 %10, %18, off offset:2048 sc0 sc1\n\tglobal_load_dwordx4 %11, %18, off offset:3072 sc0 sc1\n\t"
            "global_load_dwordx4 %12, %19, off sc0 sc1\n\tglobal_load_dwordx4 %13, %19, off offset:1024 sc0 sc1\n\t"
            "global_load_dwordx4 %14, %19, off offset:2048 sc0 sc1\n\tglobal_load_dwordx4 %15, %19, off offset:3072 sc0 sc1\n\t"
            "s_waitcnt vmcnt(0)"
            : "=&v"(o[0]), "=&v"(o[1]), "=&v"(o[2]), "=&v"(o[3]), "=&v"(o[4]), "=&v"(o[5]), "=&v"(o[6]), "=&v"(o[7]),
              "=&v"(o[8]), "=&v"(o[9]), "=&v"(o[10]), "=&v"(o[11]), "=&v"(o[12]), "=&v"(o[13]), "=&v"(o[14]), "=&v"(o[15])
            : "v"(theirs), "v"(theirs + 4096), "v"(theirs + 8192), "v"(theirs + 12288) : "memory");
#pragma unroll
        for (int f = 0; f < 16; ++f) acc[0][f >> 3][(f >> 1) & 3][f & 1] += o[f];
        const bool bad = (__hip_atomic_load(tmo, __ATOMIC_RELAXED, __HIP_MEMORY_SCOPE_AGENT) == call);
        if (d == rank) {
            store_dense(acc[0], obase + (size_t)(hh * HALF) * NTOT, bad);
        } else {
            const unsigned tot = pack_sparse(acc[0], wb, lane, bad);
            flush_sparse(wb, slot + hh * OBATCH, lane, tot);
            STAMP();
            flag_sparse(oflag + hh, lane, call, tot);
        }
    }
    STAMP();
    STAMP_END();
}

}

extern "C" void kernel_launch(int rank, int nranks, ncclComm_t comm,
                              hipStream_t stream,
                              void* const* d_in, const int* in_sizes, int n_in,
                              void* d_out, int out_size,
                              const int* in_dtypes, int out_dtype,
                              void* const* d_out_all,
                              void* const* d_ws_all, size_t ws_bytes) {
    if (nranks != NR || ws_bytes < WS_NEED) { fprintf(stderr, "unsupported configuration\n"); return; }
    static bool init = false;
    if (!init) {
        HIP_CHECK(hipFuncSetAttribute((const void*)fused_kernel, hipFuncAttributeMaxDynamicSharedMemorySize, LDS_BYTES));
        init = true;
    }
    Params p;
    p.x = (const float*)d_in[0];
    p.w = (const float*)d_in[1];
    for (int i = 0; i < NR; ++i) { p.out[i] = (float*)d_out_all[i]; p.ws[i] = (unsigned char*)d_ws_all[i]; }
    p.rank = rank;
    fused_kernel<<<256, 512, LDS_BYTES, stream>>>(p);
    HIP_CHECK(hipGetLastError());
}
```
